```python
import numpy as np

kernel_name = "gpu_gemm_rs_m4096_k4096_n4096_bf16_none"


ABSMAX_RTOL = 0.008

COMPUTE_DTYPE = "bfloat16"
EPILOGUE = "none"
IN_QUANT = None
OUT_QUANT = None
IN_DTYPES = {"x": "bfloat16", "w_mat": "bfloat16"}
WORLD_SIZE = 8

def _to_bf16(x):
    u = np.ascontiguousarray(x, dtype=np.float32).view(np.uint32)
    return ((u + 0x7FFF + ((u >> 16) & 1)) & 0xFFFF0000).view(np.float32)

def setup_inputs():

    x = _to_bf16(np.random.randn(4096, 4096).astype(np.float32))
    w_mat = _to_bf16((np.random.randn(4096, 4096) / np.sqrt(4096)).astype(np.float32))
    return {"x": x, "w_mat": w_mat}

def _epilogue(y):
    return y

def reference(x, w_mat):
    y = (x @ w_mat).astype(np.float32)
    return _epilogue(y)
```

```cpp
#include <hip/hip_runtime.h>
#include <rccl/rccl.h>
#include <cstdio>

#define HIP_CHECK(expr) do { hipError_t _e = (expr); if (_e != hipSuccess) { \
    fprintf(stderr, "HIP error %s at %s:%d\n", hipGetErrorString(_e), __FILE__, __LINE__); \
    return; } } while (0)

typedef unsigned u32;
typedef unsigned long long u64;
typedef u32 u32x4 __attribute__((ext_vector_type(4)));
typedef short s16x4 __attribute__((ext_vector_type(4)));
typedef short s16x8 __attribute__((ext_vector_type(8)));
typedef __bf16 bf16x8 __attribute__((ext_vector_type(8)));
typedef __bf16 bf16x2 __attribute__((ext_vector_type(2)));
typedef float f32x4 __attribute__((ext_vector_type(4)));
typedef float f32x2 __attribute__((ext_vector_type(2)));
#define LDSP __attribute__((address_space(3)))

constexpr int NR = 8;
constexpr u32 IN_BYTES = 4u << 20;
constexpr u32 XROW = 1024;
constexpr u32 WROW = 8192;
constexpr u32 PIECE = 8192;
constexpr u32 OFF_FX = 0;
constexpr u32 OFF_FW = 4096;
constexpr u32 OFF_FC = 65536;
constexpr u32 OFF_CTR = 131072;
constexpr u32 OFF_TMO = 196608;
constexpr u32 OFF_GATE = 200704;
constexpr u32 OFF_STAMP = 262144;
constexpr u32 OFF_DATA = 1u << 20;
constexpr u32 PAR_BYTES = 24u << 20;
constexpr u32 REL_IN = 0, REL_CP = 16u << 20;
constexpr u32 WS_NEED = OFF_DATA + 2 * PAR_BYTES;
constexpr u64 DEADLINE_TICKS = 50000000ull;
constexpr int AUX_SYS = 17;

#define PUSH_WAVES 2
#define PUSH_WIN 4
#define GATE_AT 7
#define POLL_SLEEP 2
#define C_WIN 2

struct Params {
    const char* x;
    const char* w;
    char* out;
    char* ws[NR];
    int rank;
};

template <int N> __device__ __forceinline__ void wait_vm() {
    asm volatile("s_waitcnt vmcnt(%0)" :: "i"(N) : "memory");
}
__device__ __forceinline__ u32 uni(u32 v) { return __builtin_amdgcn_readfirstlane(v); }
__device__ __forceinline__ u32x4 make_srd(const void* p, u32 bytes) {
    u64 a = (u64)p;
    u32x4 r;
    r[0] = uni((u32)a); r[1] = uni((u32)(a >> 32) & 0xffffu); r[2] = bytes; r[3] = 0x00020000u;
    return r;
}
__device__ __forceinline__ void glds4(u32 dst, u32 v0, u32 v1, u32 v2, u32 v3, u32x4 srd, u32 soff) {
    u32 keep;
    asm volatile("s_mov_b32 %0, m0\n\t"
                 "s_mov_b32 m0, %1\n\t"
                 "s_nop 4\n\t"
                 "buffer_load_dwordx4 %2, %6, %7 offen lds\n\t"
                 "buffer_load_dwordx4 %3, %6, %7 offen offset:1024 lds\n\t"
                 "buffer_load_dwordx4 %4, %6, %7 offen offset:2048 lds\n\t"
                 "buffer_load_dwordx4 %5, %6, %7 offen offset:3072 lds\n\t"
                 "s_mov_b32 m0, %0"
                 : "=&s"(keep)
                 : "s"(dst), "v"(v0), "v"(v1), "v"(v2), "v"(v3), "s"(srd), "s"(soff)
                 : "memory");
}
__device__ __forceinline__ u32 ld_sys(const char* p) {
    return __hip_atomic_load((const u32*)p, __ATOMIC_RELAXED, __HIP_MEMORY_SCOPE_SYSTEM);
}
__device__ __forceinline__ void st_sys(char* p, u32 v) {
    __hip_atomic_store((u32*)p, v, __ATOMIC_RELAXED, __HIP_MEMORY_SCOPE_SYSTEM);
}
__device__ __forceinline__ u32 ld_agent(const char* p) {
    return __hip_atomic_load((const u32*)p, __ATOMIC_RELAXED, __HIP_MEMORY_SCOPE_AGENT);
}
__device__ __forceinline__ void st_agent(char* p, u32 v) {
    __hip_atomic_store((u32*)p, v, __ATOMIC_RELAXED, __HIP_MEMORY_SCOPE_AGENT);
}
__device__ __forceinline__ u64 poll_mask(const char* p, u64 need, u32 call, char* tmo) {
    u64 t0 = 0;
    for (u32 spin = 0;; ++spin) {
        const u64 m = __ballot(ld_sys(p) >= call);
        if ((m & need) == need) return m;
        if (spin == 0) t0 = __builtin_amdgcn_s_memrealtime();
        if ((spin & 15u) == 15u) {
            if (uni(ld_agent(tmo)) == call || __builtin_amdgcn_s_memrealtime() - t0 > DEADLINE_TICKS) {
                st_agent(tmo, call);
                return ~0ull;
            }
        }
        __builtin_amdgcn_s_sleep(POLL_SLEEP);
    }
}
__device__ __forceinline__ u32 pack2(float lo, float hi) {
    return __builtin_bit_cast(u32, __builtin_convertvector(f32x2{lo, hi}, bf16x2));
}
__device__ __forceinline__ float bf_lo(u32 v) { return __builtin_bit_cast(float, v << 16); }
__device__ __forceinline__ float bf_hi(u32 v) { return __builtin_bit_cast(float, v & 0xffff0000u); }

#define STAMP(i) do { } while (0)

struct Frags { bf16x8 x[4]; bf16x8 w[4]; };

__global__ void __launch_bounds__(512, 2) gemm_rs_kernel(Params p) {
    __shared__ __attribute__((aligned(16))) char smem[8 * 16384];
    const u32 lane = threadIdx.x & 63;
    const u32 wave = uni(threadIdx.x >> 6);
    const u32 g = (u32)p.rank & 7u;
    const u32 ga = g & 1, gb = (g >> 1) & 1, gc = g >> 2;
    const u32 xcd = blockIdx.x & 7, idx = blockIdx.x >> 3;
    char* myws = p.ws[g];
    char* tmo = myws + OFF_TMO;
    char* ctr = myws + OFF_CTR + blockIdx.x * 128;
    STAMP(0);
    const u32 call_v = ld_agent(ctr) + 1;

    const bool is_p = idx < 8;
    const u32 link = xcd, lidx = idx;
    const bool has_small = !is_p && idx < 16;
    const u32 om = xcd * 8 + (idx & 7);
    const u32 o_ro = om & 3, o_cb = om >> 2;
    u32 m_rb, m_cb, m_row0, m_skip, m_dst = g, m_tile = 0;
    if (is_p) {
        const u32 other = (o_ro >> 1) ^ gb;
        m_rb = (g ^ (other * 2)) * 2 + (o_ro & 1); m_cb = o_cb; m_row0 = g * 512 + o_ro * 128; m_skip = 0;
    } else {
        const bool spill = (xcd & 5) == 0;
        const u32 d = spill ? (0x765431u >> (4 * ((xcd >> 1) * 3 + ((idx - 8) >> 3)))) & 15u : xcd;
        const u32 n = spill ? 24 + (idx & 7) : idx - 8;
        m_dst = (g ^ d) & 7u; m_skip = d;
        m_rb = m_dst * 2 + (n & 1); m_cb = n >> 1;
        m_row0 = m_dst * 512 + gb * 256 + (n & 1) * 128;
        m_tile = (n & 1) * 16 + m_cb;
    }
    const u32 T0 = has_small ? 16 : 0;
    const u32 T1 = T0 + (is_p ? 16 : 8);
    const u32 T = T1 + (is_p ? 56 : 48);

    if (wave >= 4) {
        const u32 lw = wave - 4;
        const u32 vx = (lane >> 2) * XROW + (((lane & 3) ^ ((0u - (lane >> 4)) & 3)) << 4);
        const u32 vw = (lane >> 4) * WROW + (((lane & 15) ^ ((lane >> 4) << 2)) << 4);
        if (is_p && link != 0 && lw < PUSH_WAVES) {
            const u32 dst = (g ^ link) & 7u;
            const u32 da = dst & 1, db = (dst >> 1) & 1, dc = dst >> 2;
            const u32 nx = (link == 2) ? 12 : 10, nb = nx + 16, nfirst = nb * 8, ntot = nfirst + 32;
            auto rx = __builtin_amdgcn_make_buffer_rsrc((void*)p.x, 0, IN_BYTES, 0x00020000);
            auto rw = __builtin_amdgcn_make_buffer_rsrc((void*)p.w, 0, IN_BYTES, 0x00020000);
            char* dws = p.ws[dst];
            auto rd = __builtin_amdgcn_make_buffer_rsrc((void*)dws, 0, WS_NEED, 0x00020000);
            u32x4 cur[8], nxt[8];
            u32 cur_off = 0, cur_flag = 0, nxt_off = 0, nxt_flag = 0;
            auto fetch = [&](u32 q, u32x4 (&v)[8], u32& off, u32& flag) {
                u32 j, isw, blk, row0 = 0;
                if (q < nfirst) {
                    j = q / nb;
                    const u32 e = q - j * nb;
                    isw = e >= nx;
                    if (isw) blk = e - nx;
                    else {
                        u32 n = e >> 1;
                        u32 d = (0x765431u >> (4 * n)) & 15u;
                        if (link != 2 && d >= link) d = (0x765431u >> (4 * n + 4)) & 15u;
                        const u32 slot = dst ^ d;
                        blk = slot * 2 + (e & 1);
                        row0 = slot * 512 + db * 256 + (e & 1) * 128;
                    }
                } else {
                    const u32 r = q - nfirst;
                    j = r >> 2; isw = 0;
                    const u32 other = (r >> 1) & 1;
                    blk = (dst ^ (other * 2)) * 2 + (r & 1);
                    row0 = dst * 512 + (db ^ other) * 256 + (r & 1) * 128;
                }
                const u32 k0 = da * 256 + j * 32;
                if (isw) {
                    const u32 base = k0 * WROW + (dc * 2048 + blk * 128) * 2;
#pragma unroll
                    for (int i = 0; i < 8; ++i)
                        v[i] = __builtin_amdgcn_raw_buffer_load_b128(rw, vw ^ ((i & 3) << 4), base + i * 4 * WROW, 0);
                } else {
                    const u32 base = row0 * XROW + k0 * 2;
#pragma unroll
                    for (int i = 0; i < 8; ++i)
                        v[i] = __builtin_amdgcn_raw_buffer_load_b128(rx, vx, base + i * 16 * XROW, 0);
                }
                off = REL_IN + ((g * 8 + j) * 32 + isw * 16 + blk) * PIECE;
                flag = (isw ? OFF_FW : OFF_FX) + (blk * 64 + g * 8 + j) * 4;
            };
            constexpr u32 STRIDE = 8 * PUSH_WAVES;
            const u32 q0 = lidx * PUSH_WAVES + lw;
            u32 q = q0;
            fetch(q, cur, cur_off, cur_flag);
            const u32 call = uni(call_v);
            const u32 pbase = OFF_DATA + (call & 1) * PAR_BYTES;
            u32 prev_flag = ~0u;
            for (; q < ntot; q += STRIDE) {
                const bool last = q + STRIDE >= ntot;
                fetch(last ? q : q + STRIDE, nxt, nxt_off, nxt_flag);
#pragma unroll
                for (int u = 0; u < 8; ++u) {
                    if (u < PUSH_WIN) wait_vm<PUSH_WIN - 1 + 8>();
                    else if (u < 2 * PUSH_WIN - 1) wait_vm<PUSH_WIN>();
                    else wait_vm<PUSH_WIN - 1>();
                    if (u == PUSH_WIN - 1) {
                        if (prev_flag != ~0u && lane == 0) st_sys(dws + prev_flag, call);
                    }
                    __builtin_amdgcn_raw_buffer_store_b128(cur[u], rd, lane * 16 + u * 1024, pbase + cur_off, AUX_SYS);
                    if (u == GATE_AT && last && lane == 0) st_agent(myws + OFF_GATE + link * 128 + q0 * 4, call);
                }
                prev_flag = cur_flag;
#pragma unroll
                for (int u = 0; u < 8; ++u) cur[u] = nxt[u];
                cur_off = nxt_off; cur_flag = nxt_flag;
            }
            wait_vm<0>();
            if (lane == 0) st_sys(dws + prev_flag, call);
        }
        STAMP(1);
        const u32 call = uni(call_v);
        const u32 pbase = OFF_DATA + (call & 1) * PAR_BYTES;
        const bool ldw = lw >= 2;
        const u32 lh = lw & 1;
        const u32x4 srd_loc = make_srd(ldw ? p.w : p.x, IN_BYTES);
        const u32x4 srd_ws = make_srd(myws, WS_NEED);
        u32x4 srd_null = srd_ws; srd_null[2] = 0;
        const u32 lds0 = (u32)(size_t)(LDSP char*)smem + lw * 4096;
        u32 vl[4];
#pragma unroll
        for (int i = 0; i < 4; ++i)
            vl[i] = ldw ? ((vw ^ (i << 4)) + i * (4 * WROW - 1024)) : (vx + i * (16 * XROW - 1024));
        const u32 vlin = lane * 16;
        const u32 locstep = ldw ? 32 * WROW : 64;
        const u32 kloc = is_p ? 0 : ga * 256;
        const u32 locS = ldw ? lh * 16 * WROW + ((1 - gc) * 2048 + o_cb * 128) * 2
                             : (g * 512 + o_ro * 128 + lh * 64) * XROW;
        const u32 locM = (ldw ? (kloc + lh * 16) * WROW + (gc * 2048 + m_cb * 128) * 2
                              : (m_row0 + lh * 64) * XROW + kloc * 2) - T0 * locstep;
        const u32 blk = ldw ? m_cb : m_rb;
        const u32 stg0 = pbase + REL_IN + ((ldw ? 16 : 0) + blk) * PIECE + lh * 4096;
        const char* frow = myws + (ldw ? OFF_FW : OFF_FX) + blk * 256 + lane * 4;
        u64 mask = 0;
        const u32 d_first = (m_skip == 1) ? 2 : 1;
        u32 rd_ = d_first, rj = 0;
        auto issue = [&](u32 t) {
            const u32 dstl = lds0 + (t & 7) * 16384;
            if (t < T1) {
                glds4(dstl, vl[0], vl[1], vl[2], vl[3], srd_loc, (t < T0 ? locS : locM) + t * locstep);
            } else if (t < T) {
                const u32 s = g ^ rd_;
                const u64 bit = 1ull << (s * 8 + rj);
                if (!(mask & bit)) mask = poll_mask(frow, bit, call, tmo);
                glds4(dstl, vlin, vlin, vlin, vlin, srd_ws, stg0 + (s * 8 + rj) * 32 * PIECE);
                ++rd_;
                if (rd_ == m_skip) ++rd_;
                if (rd_ >= 8) { rd_ = d_first; ++rj; }
            } else {
                glds4(dstl, vlin, vlin, vlin, vlin, srd_null, 0);
            }
        };
        wait_vm<0>();
        for (u32 t = 0; t < 6; ++t) issue(t);
        asm volatile("s_waitcnt vmcnt(20)\n\ts_barrier" ::: "memory");
        for (u32 t = 0; t < T; ++t) {
            asm volatile("s_waitcnt vmcnt(16)\n\ts_barrier" ::: "memory");
            issue(t + 6);
        }
        wait_vm<0>();
        return;
    }

    const u32 call = uni(call_v);
    const u32 pbase = OFF_DATA + (call & 1) * PAR_BYTES;
    if (wave == 0) { __builtin_amdgcn_fence(__ATOMIC_ACQUIRE, ""); wait_vm<0>(); }
    const u32 wm = wave & 1, wn = wave >> 1;
    const u32 g4 = lane >> 4, li = lane & 15;
    const u32 ax = wm * 4096 + li * 64 + ((g4 ^ ((0u - (li >> 2)) & 3)) << 4);
    u32 aw[4][2];
#pragma unroll
    for (int cc = 0; cc < 4; ++cc)
#pragma unroll
        for (int t2 = 0; t2 < 2; ++t2) {
            const u32 q = li >> 2, pp = lane & 3;
            aw[cc][t2] = 8192 + 256 * (8 * g4 + 4 * t2 + q)
                       + 16 * ((2 * (4 * wn + cc) + (pp >> 1)) ^ ((q << 2) | ((2 * g4 + t2) & 3))) + 8 * (pp & 1);
        }
    f32x4 acc[4][4], keep[4][4];
    auto readf = [&](u32 t, Frags& fr) {
        const LDSP char* sl = (const LDSP char*)smem + (t & 7) * 16384;
#pragma unroll
        for (int f = 0; f < 4; ++f) fr.x[f] = *(const LDSP bf16x8*)(sl + ax + f * 1024);
#pragma unroll
        for (int cc = 0; cc < 4; ++cc) {
            s16x4 lo = __builtin_amdgcn_ds_read_tr16_b64_v4i16((LDSP s16x4*)(sl + aw[cc][0]));
            s16x4 hi = __builtin_amdgcn_ds_read_tr16_b64_v4i16((LDSP s16x4*)(sl + aw[cc][1]));
            fr.w[cc] = __builtin_bit_cast(bf16x8, __builtin_shufflevector(lo, hi, 0, 1, 2, 3, 4, 5, 6, 7));
        }
    };
    auto mma = [&](const Frags& fr) {
#pragma unroll
        for (int c = 0; c < 4; ++c)
#pragma unroll
            for (int f = 0; f < 4; ++f)
                acc[c][f] = __builtin_amdgcn_mfma_f32_16x16x32_bf16(fr.w[c], fr.x[f], acc[c][f], 0, 0, 0);
    };
#pragma unroll
    for (int c = 0; c < 4; ++c)
#pragma unroll
        for (int f = 0; f < 4; ++f) acc[c][f] = f32x4{0.f, 0.f, 0.f, 0.f};
    Frags fa, fb;
    asm volatile("s_barrier" ::: "memory");
    readf(0, fa);
    auto steps = [&](u32 t0, u32 t1) {
        for (u32 t = t0; t < t1; t += 2) {
            asm volatile("s_waitcnt lgkmcnt(12)\n\ts_barrier" ::: "memory");
            readf(t + 1, fb);
            __builtin_amdgcn_sched_barrier(0);
            mma(fa);
            __builtin_amdgcn_sched_barrier(0);
            asm volatile("s_waitcnt lgkmcnt(12)\n\ts_barrier" ::: "memory");
            readf(t + 2, fa);
            __builtin_amdgcn_sched_barrier(0);
            mma(fb);
            __builtin_amdgcn_sched_barrier(0);
        }
    };
    steps(0, T0);
#pragma unroll
    for (int c = 0; c < 4; ++c)
#pragma unroll
        for (int f = 0; f < 4; ++f) { keep[c][f] = acc[c][f]; acc[c][f] = f32x4{0.f, 0.f, 0.f, 0.f}; }
    STAMP(2);
    steps(T0, T);
    STAMP(3);

    if (!is_p) {
        char* dws = p.ws[m_dst];
        auto rd = __builtin_amdgcn_make_buffer_rsrc((void*)dws, 0, WS_NEED, 0x00020000);
        const u32 ci = (g * 32 + m_tile) * 4 + wave;
        u32x4 pk[8];
#pragma unroll
        for (int f = 0; f < 4; ++f)
#pragma unroll
            for (int P = 0; P < 2; ++P) {
                const f32x4 lo = acc[2 * P][f], hi = acc[2 * P + 1][f];
                pk[f * 2 + P] = u32x4{pack2(lo[0], lo[1]), pack2(lo[2], lo[3]), pack2(hi[0], hi[1]), pack2(hi[2], hi[3])};
            }
        if (wave == 0) {
            constexpr u64 gm = (1ull << (8 * PUSH_WAVES)) - 1;
            poll_mask(myws + OFF_GATE + ((lane < 32 || xcd != 2) ? m_skip : 2) * 128 + (lane & 31) * 4, gm | (gm << 32), call, tmo);
        }
        STAMP(7);
        wait_vm<0>();
        for (u32 r = 0; r < 4; ++r) {
            if (r == wave) {
#pragma unroll
                for (int u = 0; u < 8; ++u) {
                    wait_vm<C_WIN - 1>();
                    __builtin_amdgcn_raw_buffer_store_b128(pk[u], rd, lane * 16 + u * 1024, pbase + REL_CP + ci * PIECE, AUX_SYS);
                }
            }
            if (r < 3) asm volatile("s_barrier" ::: "memory");
            if (r == wave) {
                wait_vm<0>();
                if (lane == 0) st_sys(dws + OFF_FC + ci * 64, call);
            }
        }
    } else {
#pragma unroll
        for (int c = 0; c < 4; ++c)
#pragma unroll
            for (int f = 0; f < 4; ++f) keep[c][f] = acc[c][f];
    }
    STAMP(4);

    if (is_p || has_small) {
        const u32 beta = o_ro >> 1;
        const u32 s0 = is_p ? ((1 - ga) | (beta << 1) | (gc << 2)) : ((beta << 1) | ((1 - gc) << 2));
        const u32 tile = (o_ro & 1) * 16 + o_cb;
        const u32 nsrc = is_p ? 1 : 2;
        auto rs = __builtin_amdgcn_make_buffer_rsrc((void*)myws, 0, WS_NEED, 0x00020000);
        for (u32 k = 0; k < nsrc; ++k) {
            const u32 ci = ((s0 + k) * 32 + tile) * 4 + wave;
            poll_mask(myws + OFF_FC + ci * 64, 1ull, call, tmo);
            u32x4 v[8];
#pragma unroll
            for (int u = 0; u < 8; ++u)
                v[u] = __builtin_amdgcn_raw_buffer_load_b128(rs, lane * 16 + u * 1024, pbase + REL_CP + ci * PIECE, 0);
#pragma unroll
            for (int f = 0; f < 4; ++f)
#pragma unroll
                for (int P = 0; P < 2; ++P) {
                    const u32x4 t = v[f * 2 + P];
                    keep[2 * P][f] += f32x4{bf_lo(t[0]), bf_hi(t[0]), bf_lo(t[1]), bf_hi(t[1])};
                    keep[2 * P + 1][f] += f32x4{bf_lo(t[2]), bf_hi(t[2]), bf_lo(t[3]), bf_hi(t[3])};
                }
        }
        STAMP(5);
        const bool bad = uni(ld_agent(tmo)) == call;
        auto ro = __builtin_amdgcn_make_buffer_rsrc((void*)p.out, 0, IN_BYTES, 0x00020000);
        const u32 ocol = (is_p ? gc : 1 - gc) * 2048 + o_cb * 128 + wn * 64;
        const u32 obase = (o_ro * 128 + wm * 64) * WROW + ocol * 2;
        const u32 vo = li * WROW + (g4 & 1) * 32 + (g4 >> 1) * 16;
#pragma unroll
        for (int f = 0; f < 4; ++f)
#pragma unroll
            for (int P = 0; P < 2; ++P) {
                const f32x4 lo = keep[2 * P][f], hi = keep[2 * P + 1][f];
                auto r0 = __builtin_amdgcn_permlane16_swap(pack2(lo[0], lo[1]), pack2(hi[0], hi[1]), false, false);
                auto r1 = __builtin_amdgcn_permlane16_swap(pack2(lo[2], lo[3]), pack2(hi[2], hi[3]), false, false);
                u32x4 o = u32x4{r0[0], r1[0], r0[1], r1[1]};
                if (bad) o = u32x4{0x7fc07fc0u, 0x7fc07fc0u, 0x7fc07fc0u, 0x7fc07fc0u};
                __builtin_amdgcn_raw_buffer_store_b128(o, ro, vo + P * 64, obase + f * 16 * WROW, 0);
            }
    }
    if (threadIdx.x == 0) st_agent(ctr, call);
    STAMP(6);
}

extern "C" void kernel_launch(int rank, int nranks, ncclComm_t comm,
                              hipStream_t stream,
                              void* const* d_in, const int* in_sizes, int n_in,
                              void* d_out, int out_size,
                              const int* in_dtypes, int out_dtype,
                              void* const* d_out_all,
                              void* const* d_ws_all, size_t ws_bytes) {
    if (nranks != NR || (unsigned)rank >= (unsigned)NR || n_in < 2 || ws_bytes < WS_NEED ||
        in_sizes[0] != 4096 * 512 || in_sizes[1] != 512 * 4096 || out_size != 512 * 4096) {
        fprintf(stderr, "kernel_launch: unexpected problem shape\n");
        return;
    }
    Params p;
    p.x = (const char*)d_in[0];
    p.w = (const char*)d_in[1];
    p.out = (char*)d_out;
    for (int i = 0; i < NR; ++i) p.ws[i] = (char*)d_ws_all[i];
    p.rank = rank;
    hipLaunchKernelGGL(gemm_rs_kernel, dim3(256), dim3(512), 0, stream, p);
    HIP_CHECK(hipGetLastError());
}
```

```cpp
#include <hip/hip_runtime.h>
#include <rccl/rccl.h>
#include <cstdio>
#include <type_traits>

#define HIP_CHECK(expr) do { hipError_t _e = (expr); if (_e != hipSuccess) { \
    fprintf(stderr, "HIP error %s at %s:%d\n", hipGetErrorString(_e), __FILE__, __LINE__); \
    return; } } while (0)

typedef unsigned u32;
typedef unsigned long long u64;
typedef u32 u32x4 __attribute__((ext_vector_type(4)));
typedef short s16x4 __attribute__((ext_vector_type(4)));
typedef short s16x8 __attribute__((ext_vector_type(8)));
typedef __bf16 bf16x8 __attribute__((ext_vector_type(8)));
typedef __bf16 bf16x2 __attribute__((ext_vector_type(2)));
typedef float f32x4 __attribute__((ext_vector_type(4)));
typedef float f32x2 __attribute__((ext_vector_type(2)));
#define LDSP __attribute__((address_space(3)))

constexpr int NR = 8;
constexpr u32 IN_BYTES = 4u << 20;
constexpr u32 XROW = 1024;
constexpr u32 WROW = 8192;
constexpr u32 PIECE = 8192;
constexpr u32 OFF_FIN = 0;
constexpr u32 OFF_FC = 65536;
constexpr u32 OFF_CTR = 131072;
constexpr u32 OFF_TMO = 196608;
constexpr u32 OFF_TOK = 200704;
constexpr u32 OFF_STAMP = 262144;
constexpr u32 OFF_DATA = 1u << 20;
constexpr u32 PAR_BYTES = 24u << 20;
constexpr u32 REL_IN = 0;
constexpr u32 REL_CP = 16u << 20;
constexpr u32 WS_NEED = OFF_DATA + 2 * PAR_BYTES;
constexpr u64 DEADLINE_TICKS = 50000000ull;
constexpr int AUX_SYS = 17;

#define PUSH_WIN 4
#define GATE_AT 7
#define POLL_SLEEP 2
#define C_WIN 2
#define C_WIN_LAST 4
#define C_LAST_POS 24
#define C_REL 5
#define G_MIN 2
#define LAG 4
#define OWN_LAG LAG
#define STG_POL 2
#define STG_AUX "sc0 sc1"
#define STG_AUXN 17
#define DEPTH 4
#define P_PACE 16
constexpr u32 NTOK = 16;
constexpr u32 C_SHARE = 2;

struct Params {
    const char* x;
    const char* w;
    char* out;
    char* self;
    char* peer[NR];
    u32 rank;
    u32 pad;
    u32 role[256];
    u64 plist[NR][NTOK][2];
};

template <int N> __device__ __forceinline__ void wait_vm() {
    asm volatile("s_waitcnt vmcnt(%0)" :: "i"(N) : "memory");
}
__device__ __forceinline__ u32 uni(u32 v) { return __builtin_amdgcn_readfirstlane(v); }
__device__ __forceinline__ u32x4 make_srd(const void* p, u32 bytes) {
    u64 a = (u64)p;
    u32x4 r;
    r[0] = uni((u32)a); r[1] = uni((u32)(a >> 32) & 0xffffu); r[2] = bytes; r[3] = 0x00020000u;
    return r;
}
#define GLDS4(POLICY)                                                                  \
    asm volatile("s_mov_b32 m0, %0\n\t"                                                \
                 "s_nop 0\n\t"                                                         \
                 "buffer_load_dwordx4 %1, %5, %6 offen " POLICY " lds\n\t"             \
                 "buffer_load_dwordx4 %2, %5, %6 offen offset:1024 " POLICY " lds\n\t" \
                 "buffer_load_dwordx4 %3, %5, %6 offen offset:2048 " POLICY " lds\n\t" \
                 "buffer_load_dwordx4 %4, %5, %6 offen offset:3072 " POLICY " lds"     \
                 :: "s"(dst), "v"(v0), "v"(v1), "v"(v2), "v"(v3), "s"(srd), "s"(soff)  \
                 : "memory")
__device__ __forceinline__ void glds4(u32 dst, u32 v0, u32 v1, u32 v2, u32 v3, u32x4 srd, u32 soff) {
    GLDS4("");
}
__device__ __forceinline__ void glds4_stg(u32 dst, u32 v0, u32x4 srd, u32 soff) {
    const u32 v1 = v0, v2 = v0, v3 = v0;
    GLDS4(STG_AUX);
}
__device__ __forceinline__ u32 ld_sys(const char* p) {
    return __hip_atomic_load((const u32*)p, __ATOMIC_RELAXED, __HIP_MEMORY_SCOPE_SYSTEM);
}
__device__ __forceinline__ void st_sys(char* p, u32 v) {
    __hip_atomic_store((u32*)p, v, __ATOMIC_RELAXED, __HIP_MEMORY_SCOPE_SYSTEM);
}
__device__ __forceinline__ u32 ld_agent(const char* p) {
    return __hip_atomic_load((const u32*)p, __ATOMIC_RELAXED, __HIP_MEMORY_SCOPE_AGENT);
}
__device__ __forceinline__ void st_agent(char* p, u32 v) {
    __hip_atomic_store((u32*)p, v, __ATOMIC_RELAXED, __HIP_MEMORY_SCOPE_AGENT);
}
__device__ __forceinline__ void add_agent(char* p, u32 v) {
    __hip_atomic_fetch_add((u32*)p, v, __ATOMIC_RELAXED, __HIP_MEMORY_SCOPE_AGENT);
}
__device__ __forceinline__ u64 poll_mask(const char* p, u64 need, u32 thr, u32 call, char* tmo) {
    u64 t0 = 0;
    for (u32 spin = 0;; ++spin) {
        const u64 m = __ballot(ld_sys(p) >= thr);
        if (m & need) return m;
        if (spin == 0) t0 = __builtin_amdgcn_s_memrealtime();
        if ((spin & 15u) == 15u) {
            if (uni(ld_agent(tmo)) == call || __builtin_amdgcn_s_memrealtime() - t0 > DEADLINE_TICKS) {
                st_agent(tmo, call);
                return ~0ull;
            }
        }
        __builtin_amdgcn_s_sleep(POLL_SLEEP);
    }
}
__device__ __forceinline__ u64 poll_any(const char* p, u64 need, u32 call, char* tmo) {
    return poll_mask(p, need, call, call, tmo);
}
__device__ __forceinline__ u32 pack2(float lo, float hi) {
    return __builtin_bit_cast(u32, __builtin_convertvector(f32x2{lo, hi}, bf16x2));
}
__device__ __forceinline__ float bf_lo(u32 v) { return __builtin_bit_cast(float, v << 16); }
__device__ __forceinline__ float bf_hi(u32 v) { return __builtin_bit_cast(float, v & 0xffff0000u); }
#define RANK_OF 0x75416320u
__host__ __device__ __forceinline__ u32 rank_of(u32 id) { return (RANK_OF >> (4 * id)) & 7u; }
__device__ __forceinline__ u32 block_row(u32 r, u32 rel, u32 h) {
    const u32 own2 = rel == 2;
    return rank_of(own2 ? r : r ^ rel) * 512 + (((r >> 1) & 1) ^ own2) * 256 + h * 128;
}

#define STAMP(i) do { } while (0)

struct Frags { bf16x8 x[4]; bf16x8 w[4]; };
constexpr u32 LDS_CTL = 8 * 16384;

__global__ void __launch_bounds__(512, 2) gemm_rs_kernel(Params p) {
    __shared__ __attribute__((aligned(16))) char smem[LDS_CTL + 64];
    const u32 lane = threadIdx.x & 63;
    const u32 wave = uni(threadIdx.x >> 6);
    const u32 g = p.rank & 7u;
    const u32 ga = g & 1, gb = (g >> 1) & 1, gc = g >> 2;
    const u32 xcd = blockIdx.x & 7, idx = blockIdx.x >> 3;
    char* myws = p.self;
    char* tmo = myws + OFF_TMO;
    char* ctr = myws + OFF_CTR + blockIdx.x * 128;
    STAMP(0);
    const u32 call_v = ld_agent(ctr);
    const bool quiet = xcd != 0 && idx < 8;
    const u32 vx = (lane >> 2) * XROW + (((lane & 3) ^ ((0u - (lane >> 4)) & 3)) << 4);
    const u32 vw = (lane >> 4) * WROW + (((lane & 15) ^ ((lane >> 4) << 2)) << 4);

    if (__builtin_expect(quiet && wave >= 6, 1)) {
        const u32 link = xcd;
        const u32 dst = g ^ link;
        u64 lst0 = p.plist[link][idx * 2 + (wave - 6)][0], lst1 = p.plist[link][idx * 2 + (wave - 6)][1];
        u32 left = link == 2 ? 16 : 15;
        auto rx = __builtin_amdgcn_make_buffer_rsrc((void*)p.x, 0, IN_BYTES, 0x00020000);
        auto rw = __builtin_amdgcn_make_buffer_rsrc((void*)p.w, 0, IN_BYTES, 0x00020000);
        char* dws = p.peer[link];
        auto rd = __builtin_amdgcn_make_buffer_rsrc((void*)dws, 0, WS_NEED, 0x00020000);
        u32x4 cur[8], nxt[8];
        u32 cur_e, nxt_e;
        auto fetch = [&](u32x4 (&v)[8], u32& e) {
            e = (u32)lst0 & 255u;
            lst0 = (lst0 >> 8) | (lst1 << 56); lst1 >>= 8;
            const u32 blk = e & 31, k0 = (dst & 1) * 256 + (e >> 5) * 32;
            if (blk >= 16) {
                const u32 base = k0 * WROW + ((dst >> 2) * 2048 + (blk - 16) * 128) * 2;
#pragma unroll
                for (int i = 0; i < 8; ++i)
                    v[i] = __builtin_amdgcn_raw_buffer_load_b128(rw, vw ^ ((i & 3) << 4), base + i * 4 * WROW, 0);
            } else {
                const u32 base = block_row(dst, blk >> 1, blk & 1) * XROW + k0 * 2;
#pragma unroll
                for (int i = 0; i < 8; ++i)
                    v[i] = __builtin_amdgcn_raw_buffer_load_b128(rx, vx, base + i * 16 * XROW, 0);
            }
        };
        fetch(cur, cur_e);
        asm volatile("s_barrier" ::: "memory");
        const u32 call = uni(call_v) + 1;
        const u32 pbase = OFF_DATA + (call & 1) * PAR_BYTES + REL_IN + link * 32 * PIECE;
        u32 prev_flag = ~0u;
        for (;;) {
            const bool last = --left == 0;
            fetch(nxt, nxt_e);
            const u32 off = pbase + ((cur_e >> 5) * 256 + (cur_e & 31)) * PIECE;
#pragma unroll
            for (int u = 0; u < 8; ++u) {
                if (u < PUSH_WIN) wait_vm<PUSH_WIN - 1 + 8>();
                else if (u < 2 * PUSH_WIN - 1) wait_vm<PUSH_WIN>();
                else wait_vm<PUSH_WIN - 1>();
                if (u == PUSH_WIN - 1 && prev_flag != ~0u && lane == 0) st_sys(dws + prev_flag, call);
                __builtin_amdgcn_raw_buffer_store_b128(cur[u], rd, lane * 16 + u * 1024, off, AUX_SYS);
                if (u == GATE_AT && last && lane == 0) add_agent(myws + OFF_TOK + link * 256, 1);
            }
            prev_flag = OFF_FIN + ((cur_e & 31) * 64 + (cur_e >> 5) * 8 + link) * 4;
            if (last) break;
#pragma unroll
            for (int u = 0; u < 8; ++u) cur[u] = nxt[u];
            cur_e = nxt_e;
        }
        ((volatile LDSP u32*)(smem + LDS_CTL))[wave + 2] = 1;
        wait_vm<0>();
        if (lane == 0) st_sys(dws + prev_flag, call);
        STAMP(1);
        return;
    }

    const u32 role = p.role[blockIdx.x];
    const u32 m_d = role & 7, m_h = (role >> 3) & 1, m_cb = (role >> 4) & 15;
    const bool is_p = (m_d & 5) == 0;
    const bool has_small = (role >> 13) & 1;
    const u32 o_ro = is_p ? ((gb ^ (m_d >> 1)) << 1) | m_h : (role >> 14) & 3;
    const u32 o_cb = is_p ? m_cb : (role >> 16) & 15;
    const u32 T0 = has_small ? 16 : 0;
    const u32 T1 = T0 + (is_p ? 16 : 8);
    const u32 T = T1 + (is_p ? 56 : 48);
    volatile LDSP u32* ctl = (volatile LDSP u32*)(smem + LDS_CTL);
    if (wave == 0 && lane < 16) ctl[lane] = 0;

    if (wave >= 4) {
        const bool ldw = wave & 1;
        const u32 lh = (wave >> 1) & 1;
        const u32x4 srd_loc = make_srd(ldw ? p.w : p.x, IN_BYTES);
        const u32x4 srd_ws = make_srd(myws, WS_NEED);
        asm volatile("s_nop 4" :: "s"(srd_loc), "s"(srd_ws));
        const u32 lds0 = (u32)(size_t)(LDSP char*)smem + (ldw ? 8192 : 0) + lh * 4096;
        u32 vl[4];
#pragma unroll
        for (int i = 0; i < 4; ++i)
            vl[i] = ldw ? ((vw ^ (i << 4)) + i * (4 * WROW - 1024)) : (vx + i * (16 * XROW - 1024));
        const u32 vlin = lane * 16;
        const u32 locstep = ldw ? 32 * WROW : 64;
        const u32 lochalf = ldw ? 16 * WROW : 64 * XROW;
        const u32 kloc = is_p ? 0 : ga * 256;
        const u32 locS = (ldw ? ((1 - gc) * 2048 + o_cb * 128) * 2 : (rank_of(g) * 512 + o_ro * 128) * XROW) + lh * lochalf;
        const u32 locM = (ldw ? kloc * WROW + (gc * 2048 + m_cb * 128) * 2 : block_row(g, m_d, m_h) * XROW + kloc * 2)
                       + lh * lochalf - T0 * locstep;
        auto issue_loc = [&](u32 t, u32 half) {
            glds4(lds0 + (t & 7) * 16384 + half * 4096, vl[0], vl[1], vl[2], vl[3], srd_loc,
                  (t < T0 ? locS : locM) + t * locstep + half * lochalf);
        };
        volatile LDSP u32* full = ctl + (wave - 4);
        const u32 call = uni(call_v) + 1;
        const u32 blk = ldw ? 16 + m_cb : m_d * 2 + m_h;
        const u32 stg0 = OFF_DATA + (call & 1) * PAR_BYTES + REL_IN + blk * PIECE + lh * 4096;
        const char* frow = myws + OFF_FIN + blk * 256 + lane * 4;
        auto stage = [&](auto halves) {
            constexpr int H = decltype(halves)::value, V = 4 * H;
            u32 pub = 0, lim = 8, t = 0;
            auto set_full = [&](u32 v) { full[0] = v; if (H == 2) full[2] = v; };
            auto publish1 = [&](auto extra) {
                constexpr int E = decltype(extra)::value;
                const u32 n = t - pub;
                if (n >= 6) wait_vm<5 * V + E>();
                else if (n == 5) wait_vm<4 * V + E>();
                else if (n == 4) wait_vm<3 * V + E>();
                else if (n == 3) wait_vm<2 * V + E>();
                else if (n == 2) wait_vm<V + E>();
                else wait_vm<E>();
                set_full(++pub);
            };
            using E0 = std::integral_constant<int, 0>;
            using E1 = std::integral_constant<int, 1>;
            auto make_room = [&]() {
                if (__builtin_expect(t - pub >= DEPTH, 1)) { wait_vm<V * (DEPTH - 1)>(); set_full(++pub); }
                while (__builtin_expect(t >= lim, 0)) {
                    const u32x4 dn = *(volatile LDSP u32x4*)(ctl + 4);
                    lim = uni(min(min(dn[0], dn[1]), min(dn[2], dn[3]))) + 8;
                    if (t < lim) break;
                    if (pub < t) publish1(E0{}); else __builtin_amdgcn_s_sleep(1);
                }
            };
            if (H == 1) for (; t < DEPTH; ++t) issue_loc(t, 0);
            asm volatile("s_waitcnt lgkmcnt(0)\n\ts_barrier" ::: "memory");
            for (; t < T1; ++t) {
                if (H == 2) __builtin_amdgcn_s_sleep(P_PACE);
                make_room();
                issue_loc(t, 0);
                if (H == 2) issue_loc(t, 1);
            }
            u64 mask = 0;
            u64 todo = 0xfefefefefefefefeull & ~(0x0101010101010101ull << (is_p ? 0 : m_d));
            bool slow = H == 2;
            do {
                if (slow) {
                    while (pub < t) publish1(E0{});
                    slow = uni(ctl[8] + ctl[9]) < 2;
                }
                make_room();
                const u32 b = __builtin_ctzll(todo);
                todo &= todo - 1;
                if (__builtin_expect(!((mask >> b) & 1), 0)) {
                    const u32 fv = ld_sys(frow);
                    while (pub < t) publish1(E1{});
                    mask = __ballot(fv >= call);
                    if (!((mask >> b) & 1)) mask = poll_mask(frow, 1ull << b, call, call, tmo);
                }
                glds4_stg(lds0 + (t & 7) * 16384, vlin, srd_ws, stg0 + b * 32 * PIECE);
                if (H == 2) glds4_stg(lds0 + (t & 7) * 16384 + 4096, vlin, srd_ws, stg0 + b * 32 * PIECE + 4096);
                ++t;
            } while (todo);
            while (pub < t) publish1(E0{});
            set_full(0x7fffffffu);
        };
        if (quiet) stage(std::integral_constant<int, 2>{});
        else stage(std::integral_constant<int, 1>{});
        return;
    }

    const u32 call = uni(call_v) + 1;
    const u32 cbase = OFF_DATA + (call & 1) * PAR_BYTES + REL_CP;
    const u32 wm = wave & 1, wn = wave >> 1;
    const u32 g4 = lane >> 4, li = lane & 15;
    const u32 ax = wm * 4096 + li * 64 + ((g4 ^ ((0u - (li >> 2)) & 3)) << 4);
    u32 aw[4][2];
#pragma unroll
    for (int cc = 0; cc < 4; ++cc)
#pragma unroll
        for (int t2 = 0; t2 < 2; ++t2) {
            const u32 q = li >> 2, pp = lane & 3;
            aw[cc][t2] = 8192 + 256 * (8 * g4 + 4 * t2 + q)
                       + 16 * ((2 * (4 * wn + cc) + (pp >> 1)) ^ ((q << 2) | ((2 * g4 + t2) & 3))) + 8 * (pp & 1);
        }
    f32x4 acc[4][4], keep[4][4];
    auto readf = [&](u32 t, Frags& fr) {
        const LDSP char* sl = (const LDSP char*)smem + (t & 7) * 16384;
#pragma unroll
        for (int f = 0; f < 4; ++f) fr.x[f] = *(const LDSP bf16x8*)(sl + ax + f * 1024);
#pragma unroll
        for (int cc = 0; cc < 4; ++cc) {
            s16x4 lo = __builtin_amdgcn_ds_read_tr16_b64_v4i16((LDSP s16x4*)(sl + aw[cc][0]));
            s16x4 hi = __builtin_amdgcn_ds_read_tr16_b64_v4i16((LDSP s16x4*)(sl + aw[cc][1]));
            fr.w[cc] = __builtin_bit_cast(bf16x8, __builtin_shufflevector(lo, hi, 0, 1, 2, 3, 4, 5, 6, 7));
        }
    };
    auto mma = [&](const Frags& fr) {
#pragma unroll
        for (int c = 0; c < 4; ++c)
#pragma unroll
            for (int f = 0; f < 4; ++f)
                acc[c][f] = __builtin_amdgcn_mfma_f32_16x16x32_bf16(fr.w[c], fr.x[f], acc[c][f], 0, 0, 0);
    };
    auto interleave = [&]() {
#pragma unroll
        for (int i = 0; i < 4; ++i) {
            __builtin_amdgcn_sched_group_barrier(0x8, 1, 0);
            __builtin_amdgcn_sched_group_barrier(0x2, 2, 0);
            __builtin_amdgcn_sched_group_barrier(0x100, 2, 0);
        }
#pragma unroll
        for (int i = 0; i < 4; ++i) {
            __builtin_amdgcn_sched_group_barrier(0x8, 1, 0);
            __builtin_amdgcn_sched_group_barrier(0x2, 1, 0);
            __builtin_amdgcn_sched_group_barrier(0x100, 1, 0);
        }
        __builtin_amdgcn_sched_group_barrier(0x8, 8, 0);
        __builtin_amdgcn_sched_barrier(0);
    };
#pragma unroll
    for (int c = 0; c < 4; ++c)
#pragma unroll
        for (int f = 0; f < 4; ++f) acc[c][f] = f32x4{0.f, 0.f, 0.f, 0.f};
    u32 avail = 0;
    auto need = [&](u32 t) {
        while (__builtin_expect(avail <= t, 0)) {
            const u32x4 f = *(volatile LDSP u32x4*)ctl;
            avail = uni(min(min(f[0], f[1]), min(f[2], f[3])));
            if (avail <= t) __builtin_amdgcn_s_sleep(1);
        }
        asm volatile("" ::: "memory");
    };
    auto retire = [&](u32 t) {
        asm volatile("s_waitcnt lgkmcnt(0)" ::: "memory");
        ctl[4 + wave] = t + 1;
    };
    Frags fa, fb;
    asm volatile("s_waitcnt lgkmcnt(0)\n\ts_barrier" ::: "memory");
    need(0);
    readf(0, fa);
    auto steps = [&](u32 t0, u32 t1) {
        for (u32 t = t0; t < t1; t += 2) {
            retire(t);
            need(t + 1);
            readf(t + 1, fb);
            mma(fa);
            interleave();
            retire(t + 1);
            need(t + 2);
            readf(t + 2, fa);
            mma(fb);
            interleave();
        }
    };
    steps(0, T0);
#pragma unroll
    for (int c = 0; c < 4; ++c)
#pragma unroll
        for (int f = 0; f < 4; ++f) { keep[c][f] = acc[c][f]; acc[c][f] = f32x4{0.f, 0.f, 0.f, 0.f}; }
    STAMP(2);
    steps(T0, T);
    STAMP(3);

    if (!is_p) {
        char* dws = p.peer[m_d];
        auto rd = __builtin_amdgcn_make_buffer_rsrc((void*)dws, 0, WS_NEED, 0x00020000);
        const u32 ci = (m_d * 32 + m_h * 16 + m_cb) * 4 + wave;
        u32x4 pk[8];
#pragma unroll
        for (int f = 0; f < 4; ++f)
#pragma unroll
            for (int P = 0; P < 2; ++P) {
                const f32x4 lo = acc[2 * P][f], hi = acc[2 * P + 1][f];
                pk[f * 2 + P] = u32x4{pack2(lo[0], lo[1]), pack2(lo[2], lo[3]), pack2(hi[0], hi[1]), pack2(hi[2], hi[3])};
            }
        char* tok = myws + OFF_TOK + m_d * 256;
        const u32 pos = (role >> 8) & 31;
        if (wave == 0)
            poll_mask(tok, 1ull, (call - 1) * (NTOK + 32 * C_SHARE) + max((pos + 1) * C_SHARE, (u32)G_MIN), call, tmo);
        asm volatile("s_waitcnt vmcnt(0)\n\ts_barrier" ::: "memory");
        STAMP(7);
        const bool wide = pos >= C_LAST_POS;
#pragma unroll
        for (int u = 0; u < 8; ++u) {
            if (wide) wait_vm<C_WIN_LAST - 1>(); else wait_vm<C_WIN - 1>();
            __builtin_amdgcn_raw_buffer_store_b128(pk[u], rd, lane * 16 + u * 1024, cbase + ci * PIECE, AUX_SYS);
            if (u == C_REL && wave == 0 && lane == 0) add_agent(tok, C_SHARE);
        }
        wait_vm<0>();
        if (lane == 0) st_sys(dws + OFF_FC + ci * 64, call);
    } else {
#pragma unroll
        for (int c = 0; c < 4; ++c)
#pragma unroll
            for (int f = 0; f < 4; ++f) keep[c][f] = acc[c][f];
    }
    STAMP(4);

    if (is_p || has_small) {
        const u32 db = ((o_ro >> 1) ^ gb) << 1;
        const u32 l0 = is_p ? (1 | db) : (4 | db);
        const u32 tile = (o_ro & 1) * 16 + o_cb;
        auto rs = __builtin_amdgcn_make_buffer_rsrc((void*)myws, 0, WS_NEED, 0x00020000);
        const u32 ci0 = (l0 * 32 + tile) * 4 + wave;
        u32 left = is_p ? 1 : 3, tmo_v = 0;
        while (left) {
            const u32 got = (u32)poll_any(myws + OFF_FC + (ci0 + (lane & 1) * 128) * 64, left, call, tmo) & left;
            const u32 k = (got & 1) ? 0 : 1;
            left &= ~(1u << k);
            u32x4 v[8];
#pragma unroll
            for (int u = 0; u < 8; ++u)
                v[u] = __builtin_amdgcn_raw_buffer_load_b128(rs, lane * 16 + u * 1024, cbase + (ci0 + k * 128) * PIECE, STG_AUXN);
            tmo_v = ld_agent(tmo);
#pragma unroll
            for (int f = 0; f < 4; ++f)
#pragma unroll
                for (int P = 0; P < 2; ++P) {
                    const u32x4 t = v[f * 2 + P];
                    keep[2 * P][f] += f32x4{bf_lo(t[0]), bf_hi(t[0]), bf_lo(t[1]), bf_hi(t[1])};
                    keep[2 * P + 1][f] += f32x4{bf_lo(t[2]), bf_hi(t[2]), bf_lo(t[3]), bf_hi(t[3])};
                }
        }
        STAMP(5);
        const bool bad = uni(tmo_v) == call;
        auto ro = __builtin_amdgcn_make_buffer_rsrc((void*)p.out, 0, IN_BYTES, 0x00020000);
        const u32 ocol = (is_p ? gc : 1 - gc) * 2048 + o_cb * 128 + wn * 64;
        const u32 obase = (o_ro * 128 + wm * 64) * WROW + ocol * 2;
        const u32 vo = li * WROW + (g4 & 1) * 32 + (g4 >> 1) * 16;
#pragma unroll
        for (int f = 0; f < 4; ++f)
#pragma unroll
            for (int P = 0; P < 2; ++P) {
                const f32x4 lo = keep[2 * P][f], hi = keep[2 * P + 1][f];
                auto r0 = __builtin_amdgcn_permlane16_swap(pack2(lo[0], lo[1]), pack2(hi[0], hi[1]), false, false);
                auto r1 = __builtin_amdgcn_permlane16_swap(pack2(lo[2], lo[3]), pack2(hi[2], hi[3]), false, false);
                u32x4 o = u32x4{r0[0], r1[0], r0[1], r1[1]};
                if (bad) o = u32x4{0x7fc07fc0u, 0x7fc07fc0u, 0x7fc07fc0u, 0x7fc07fc0u};
                __builtin_amdgcn_raw_buffer_store_b128(o, ro, vo + P * 64, obase + f * 16 * WROW, 0);
            }
    }
    if (threadIdx.x == 0) st_agent(ctr, call);
    STAMP(6);
}

static void build_schedule(Params& p) {
    static const int CL[6] = {1, 3, 4, 5, 6, 7};
    int nsmall = 0;
    for (int n = 0; n < 64; ++n)
        p.role[(n & 31) * 8 + (n >> 5) * 2] = (u32)(((n >> 1) & 1) * 2 | (n & 1) << 3 | (n >> 2) << 4);
    for (int pos = 0; pos < 32; ++pos)
        for (int li = 0; li < 6; ++li) {
            u32 e = (u32)(CL[li] | (pos & 1) << 3 | (pos >> 1) << 4 | pos << 8);
            if (nsmall < 64) { e |= 1u << 13 | (u32)nsmall << 14; ++nsmall; }
            p.role[((pos + 8) & 31) * 8 + CL[li]] = e;
        }
    for (int l = 0; l < 8; ++l) {
        unsigned char lst[256] = {0};
        int n = 0;
        if (l) {
            for (int r = 0; r < 8 + LAG; ++r) {
                if (r < 8) {
                    for (int li = 0; li < 6; ++li)
                        if (CL[li] != l) { lst[n++] = (unsigned char)(CL[li] * 2 | r << 5); lst[n++] = (unsigned char)((CL[li] * 2 + 1) | r << 5); }
                    for (int c = 0; c < 8; ++c) lst[n++] = (unsigned char)((16 + c) | r << 5);
                }
                if (r >= OWN_LAG && r < 8 + OWN_LAG)
                    for (int k = 0; k < 4; ++k) lst[n++] = (unsigned char)(((k >> 1) * 4 + (k & 1)) | (r - OWN_LAG) << 5);
                if (r >= LAG)
                    for (int c = 8; c < 16; ++c) lst[n++] = (unsigned char)((16 + c) | (r - LAG) << 5);
            }
        }
        for (int w = 0; w < (int)NTOK; ++w) {
            u64 v[2] = {0, 0};
            for (int k = 0; k < 16; ++k) v[k >> 3] |= (u64)lst[w + 16 * k] << (8 * (k & 7));
            p.plist[l][w][0] = v[0]; p.plist[l][w][1] = v[1];
        }
    }
}

extern "C" void kernel_launch(int rank, int nranks, ncclComm_t comm,
                              hipStream_t stream,
                              void* const* d_in, const int* in_sizes, int n_in,
                              void* d_out, int out_size,
                              const int* in_dtypes, int out_dtype,
                              void* const* d_out_all,
                              void* const* d_ws_all, size_t ws_bytes) {
    if (nranks != NR || (unsigned)rank >= (unsigned)NR || n_in < 2 || ws_bytes < WS_NEED ||
        in_sizes[0] != 4096 * 512 || in_sizes[1] != 512 * 4096 || out_size != 512 * 4096) {
        fprintf(stderr, "kernel_launch: unexpected problem shape\n");
        return;
    }
    Params p;
    p.x = (const char*)d_in[0];
    p.w = (const char*)d_in[1];
    p.out = (char*)d_out;
    u32 id = 0;
    while (rank_of(id) != (u32)rank) ++id;
    for (u32 d = 0; d < NR; ++d) p.peer[d] = (char*)d_ws_all[rank_of(id ^ d)];
    p.self = p.peer[0];
    p.rank = id;
    p.pad = 0;
    build_schedule(p);
    hipLaunchKernelGGL(gemm_rs_kernel, dim3(256), dim3(512), 0, stream, p);
    HIP_CHECK(hipGetLastError());
}
```

```cpp
#include <hip/hip_runtime.h>
#include <rccl/rccl.h>
#include <cstdio>
#include <type_traits>

#define HIP_CHECK(expr) do { hipError_t _e = (expr); if (_e != hipSuccess) { \
    fprintf(stderr, "HIP error %s at %s:%d\n", hipGetErrorString(_e), __FILE__, __LINE__); \
    return; } } while (0)

typedef unsigned u32;
typedef unsigned long long u64;
typedef u32 u32x4 __attribute__((ext_vector_type(4)));
typedef short s16x4 __attribute__((ext_vector_type(4)));
typedef short s16x8 __attribute__((ext_vector_type(8)));
typedef __bf16 bf16x8 __attribute__((ext_vector_type(8)));
typedef __bf16 bf16x2 __attribute__((ext_vector_type(2)));
typedef float f32x4 __attribute__((ext_vector_type(4)));
typedef float f32x2 __attribute__((ext_vector_type(2)));
#define LDSP __attribute__((address_space(3)))

constexpr int NR = 8;
constexpr u32 IN_BYTES = 4u << 20;
constexpr u32 XROW = 1024;
constexpr u32 WROW = 8192;
constexpr u32 PIECE = 8192;
constexpr u32 OFF_FIN = 0;
constexpr u32 OFF_FC = 65536;
constexpr u32 OFF_CTR = 131072;
constexpr u32 OFF_CLAIM = 163840;
constexpr u32 OFF_TMO = 196608;
constexpr u32 OFF_TOK = 200704;
constexpr u32 OFF_STAMP = 262144;
constexpr u32 OFF_DATA = 1u << 20;
constexpr u32 PAR_BYTES = 24u << 20;
constexpr u32 REL_IN = 0;
constexpr u32 REL_CP = 16u << 20;
constexpr u32 WS_NEED = OFF_DATA + 2 * PAR_BYTES;
constexpr u64 DEADLINE_TICKS = 50000000ull;
constexpr int AUX_SYS = 17;

#define PUSH_WIN 4
#define GATE_AT 7
#define POLL_SLEEP 2
#define C_WIN 2
#define C_WIN_LAST 4
#define C_LAST_POS 24
#define C_REL 5
#define G_MIN 2
#define LAG 4
#define OWN_LAG LAG
#define STG_POL 2
#define STG_AUX "sc0 sc1"
#define STG_AUXN 17
#define DEPTH 4
#define P_PACE 16
constexpr u32 NTOK = 16;
constexpr u32 C_SHARE = 2;

struct Params {
    const char* x;
    const char* w;
    char* out;
    char* self;
    char* peer[NR];
    u32 rank;
    u32 pad;
    u32 role[256];
    u64 plist[NR][NTOK][2];
};

template <int N> __device__ __forceinline__ void wait_vm() {
    asm volatile("s_waitcnt vmcnt(%0)" :: "i"(N) : "memory");
}
__device__ __forceinline__ u32 uni(u32 v) { return __builtin_amdgcn_readfirstlane(v); }
__device__ __forceinline__ u32x4 make_srd(const void* p, u32 bytes) {
    u64 a = (u64)p;
    u32x4 r;
    r[0] = uni((u32)a); r[1] = uni((u32)(a >> 32) & 0xffffu); r[2] = bytes; r[3] = 0x00020000u;
    return r;
}
#define GLDS4(POLICY)                                                                  \
    asm volatile("s_mov_b32 m0, %0\n\t"                                                \
                 "s_nop 0\n\t"                                                         \
                 "buffer_load_dwordx4 %1, %5, %6 offen " POLICY " lds\n\t"             \
                 "buffer_load_dwordx4 %2, %5, %6 offen offset:1024 " POLICY " lds\n\t" \
                 "buffer_load_dwordx4 %3, %5, %6 offen offset:2048 " POLICY " lds\n\t" \
                 "buffer_load_dwordx4 %4, %5, %6 offen offset:3072 " POLICY " lds"     \
                 :: "s"(dst), "v"(v0), "v"(v1), "v"(v2), "v"(v3), "s"(srd), "s"(soff)  \
                 : "memory")
__device__ __forceinline__ void glds4(u32 dst, u32 v0, u32 v1, u32 v2, u32 v3, u32x4 srd, u32 soff) {
    GLDS4("");
}
__device__ __forceinline__ void glds4_stg(u32 dst, u32 v0, u32x4 srd, u32 soff) {
    const u32 v1 = v0, v2 = v0, v3 = v0;
    GLDS4(STG_AUX);
}
__device__ __forceinline__ u32 ld_sys(const char* p) {
    return __hip_atomic_load((const u32*)p, __ATOMIC_RELAXED, __HIP_MEMORY_SCOPE_SYSTEM);
}
__device__ __forceinline__ void st_sys(char* p, u32 v) {
    __hip_atomic_store((u32*)p, v, __ATOMIC_RELAXED, __HIP_MEMORY_SCOPE_SYSTEM);
}
__device__ __forceinline__ u32 ld_agent(const char* p) {
    return __hip_atomic_load((const u32*)p, __ATOMIC_RELAXED, __HIP_MEMORY_SCOPE_AGENT);
}
__device__ __forceinline__ void st_agent(char* p, u32 v) {
    __hip_atomic_store((u32*)p, v, __ATOMIC_RELAXED, __HIP_MEMORY_SCOPE_AGENT);
}
__device__ __forceinline__ void add_agent(char* p, u32 v) {
    __hip_atomic_fetch_add((u32*)p, v, __ATOMIC_RELAXED, __HIP_MEMORY_SCOPE_AGENT);
}
__device__ __forceinline__ u64 poll_mask(const char* p, u64 need, u32 thr, u32 call, char* tmo) {
    u64 t0 = 0;
    for (u32 spin = 0;; ++spin) {
        const u64 m = __ballot(ld_sys(p) >= thr);
        if (m & need) return m;
        if (spin == 0) t0 = __builtin_amdgcn_s_memrealtime();
        if ((spin & 15u) == 15u) {
            if (uni(ld_agent(tmo)) == call || __builtin_amdgcn_s_memrealtime() - t0 > DEADLINE_TICKS) {
                st_agent(tmo, call);
                return ~0ull;
            }
        }
        __builtin_amdgcn_s_sleep(POLL_SLEEP);
    }
}
__device__ __forceinline__ u64 poll_any(const char* p, u64 need, u32 call, char* tmo) {
    return poll_mask(p, need, call, call, tmo);
}
__device__ __forceinline__ u32 pack2(float lo, float hi) {
    return __builtin_bit_cast(u32, __builtin_convertvector(f32x2{lo, hi}, bf16x2));
}
__device__ __forceinline__ float bf_lo(u32 v) { return __builtin_bit_cast(float, v << 16); }
__device__ __forceinline__ float bf_hi(u32 v) { return __builtin_bit_cast(float, v & 0xffff0000u); }
#define RANK_OF 0x75416320u
__host__ __device__ __forceinline__ u32 rank_of(u32 id) { return (RANK_OF >> (4 * id)) & 7u; }
__device__ __forceinline__ u32 block_row(u32 r, u32 rel, u32 h) {
    const u32 own2 = rel == 2;
    return rank_of(own2 ? r : r ^ rel) * 512 + (((r >> 1) & 1) ^ own2) * 256 + h * 128;
}

#define STAMP(i) do { } while (0)

struct Frags { bf16x8 x[4]; bf16x8 w[4]; };
constexpr u32 LDS_CTL = 8 * 16384;

__global__ void __launch_bounds__(512, 2) gemm_rs_kernel(Params p) {
    __shared__ __attribute__((aligned(16))) char smem[LDS_CTL + 64];
    const u32 lane = threadIdx.x & 63;
    const u32 wave = uni(threadIdx.x >> 6);
    const u32 g = p.rank & 7u;
    const u32 ga = g & 1, gb = (g >> 1) & 1, gc = g >> 2;
    char* myws = p.self;
    char* tmo = myws + OFF_TMO;
    char* ctr = myws + OFF_CTR + blockIdx.x * 128;
    volatile LDSP u32* ctl = (volatile LDSP u32*)(smem + LDS_CTL);
    STAMP(0);
    u32 item = uni((blockIdx.x & ~7u) | (__builtin_amdgcn_s_getreg((3 << 11) | 20) & 7u));
    if (wave == 0) {
        if (lane == 0) {
            u32* claim = (u32*)(myws + OFF_CLAIM);
            const u32 old = __hip_atomic_fetch_add(claim + item * 32, 1u, __ATOMIC_RELAXED, __HIP_MEMORY_SCOPE_AGENT);
            const u32 calls = ld_agent(ctr);
            u32 it = item;
            if (__builtin_expect(old != calls, 0)) {
                __hip_atomic_fetch_add(claim + item * 32, ~0u, __ATOMIC_RELAXED, __HIP_MEMORY_SCOPE_AGENT);
#pragma clang loop unroll(disable)
                for (u32 tries = 0;; ++tries) {
                    it = (it + 1) & 255u;
                    u32 expect = calls;
                    if (__hip_atomic_compare_exchange_strong(claim + it * 32, &expect, calls + 1, __ATOMIC_RELAXED,
                                                             __ATOMIC_RELAXED, __HIP_MEMORY_SCOPE_AGENT)) break;
                    if (tries > 4096) { st_agent(tmo, calls + 1); break; }
                }
            }
            ctl[10] = it; ctl[11] = calls + 1;
        } else if (lane <= 10) {
            ctl[lane - 1] = 0;
        }
    }
    const u32 vx = (lane >> 2) * XROW + (((lane & 3) ^ ((0u - (lane >> 4)) & 3)) << 4);
    const u32 vw = (lane >> 4) * WROW + (((lane & 15) ^ ((lane >> 4) << 2)) << 4);
    auto pushes = [](u32 it) { return (it & 7) != 0 && it < 64; };
    auto rx = __builtin_amdgcn_make_buffer_rsrc((void*)p.x, 0, IN_BYTES, 0x00020000);
    auto rw = __builtin_amdgcn_make_buffer_rsrc((void*)p.w, 0, IN_BYTES, 0x00020000);
    u64 lst0 = 0, lst1 = 0;
    u32x4 cur[8];
    u32 cur_e = 0;
    auto fetch = [&](u32x4 (&v)[8], u32& e) {
        const u32 dst = g ^ (item & 7);
        e = (u32)lst0 & 255u;
        lst0 = (lst0 >> 8) | (lst1 << 56); lst1 >>= 8;
        const u32 blk = e & 31, k0 = (dst & 1) * 256 + (e >> 5) * 32;
        if (blk >= 16) {
            const u32 base = k0 * WROW + ((dst >> 2) * 2048 + (blk - 16) * 128) * 2;
#pragma unroll
            for (int i = 0; i < 8; ++i)
                v[i] = __builtin_amdgcn_raw_buffer_load_b128(rw, vw ^ ((i & 3) << 4), base + i * 4 * WROW, 0);
        } else {
            const u32 base = block_row(dst, blk >> 1, blk & 1) * XROW + k0 * 2;
#pragma unroll
            for (int i = 0; i < 8; ++i)
                v[i] = __builtin_amdgcn_raw_buffer_load_b128(rx, vx, base + i * 16 * XROW, 0);
        }
    };
    auto first_piece = [&]() {
        lst0 = p.plist[item & 7][(item >> 3) * 2 + (wave - 6)][0];
        lst1 = p.plist[item & 7][(item >> 3) * 2 + (wave - 6)][1];
        fetch(cur, cur_e);
    };
    if (wave >= 6 && pushes(item)) first_piece();
    asm volatile("s_waitcnt lgkmcnt(0)\n\ts_barrier" ::: "memory");
    const u32 call = uni(ctl[11]);
    {
        const u32 it = uni(ctl[10]);
        if (__builtin_expect(it != item, 0)) {
            item = it;
            if (wave >= 6 && pushes(item)) first_piece();
        }
    }
    const bool quiet = pushes(item);

    if (__builtin_expect(quiet && wave >= 6, 1)) {
        const u32 link = item & 7;
        u32 left = link == 2 ? 16 : 15;
        char* dws = p.peer[link];
        auto rd = __builtin_amdgcn_make_buffer_rsrc((void*)dws, 0, WS_NEED, 0x00020000);
        u32x4 nxt[8];
        u32 nxt_e;
        const u32 pbase = OFF_DATA + (call & 1) * PAR_BYTES + REL_IN + link * 32 * PIECE;
        u32 prev_flag = ~0u;
        for (;;) {
            const bool last = --left == 0;
            fetch(nxt, nxt_e);
            const u32 off = pbase + ((cur_e >> 5) * 256 + (cur_e & 31)) * PIECE;
#pragma unroll
            for (int u = 0; u < 8; ++u) {
                if (u < PUSH_WIN) wait_vm<PUSH_WIN - 1 + 8>();
                else if (u < 2 * PUSH_WIN - 1) wait_vm<PUSH_WIN>();
                else wait_vm<PUSH_WIN - 1>();
                if (u == PUSH_WIN - 1 && prev_flag != ~0u && lane == 0) st_sys(dws + prev_flag, call);
                __builtin_amdgcn_raw_buffer_store_b128(cur[u], rd, lane * 16 + u * 1024, off, AUX_SYS);
                if (u == GATE_AT && last && lane == 0) add_agent(myws + OFF_TOK + link * 256, 1);
            }
            prev_flag = OFF_FIN + ((cur_e & 31) * 64 + (cur_e >> 5) * 8 + link) * 4;
            if (last) break;
#pragma unroll
            for (int u = 0; u < 8; ++u) cur[u] = nxt[u];
            cur_e = nxt_e;
        }
        ctl[wave + 2] = 1;
        wait_vm<0>();
        if (lane == 0) st_sys(dws + prev_flag, call);
        STAMP(1);
        return;
    }

    const u32 role = p.role[item];
    const u32 m_d = role & 7, m_h = (role >> 3) & 1, m_cb = (role >> 4) & 15;
    const bool is_p = (m_d & 5) == 0;
    const bool has_small = (role >> 13) & 1;
    const u32 o_ro = is_p ? ((gb ^ (m_d >> 1)) << 1) | m_h : (role >> 14) & 3;
    const u32 o_cb = is_p ? m_cb : (role >> 16) & 15;
    const u32 T0 = has_small ? 16 : 0;
    const u32 T1 = T0 + (is_p ? 16 : 8);
    const u32 T = T1 + (is_p ? 56 : 48);

    if (wave >= 4) {
        const bool ldw = wave & 1;
        const u32 lh = (wave >> 1) & 1;
        const u32x4 srd_loc = make_srd(ldw ? p.w : p.x, IN_BYTES);
        const u32x4 srd_ws = make_srd(myws, WS_NEED);
        asm volatile("s_nop 4" :: "s"(srd_loc), "s"(srd_ws));
        const u32 lds0 = (u32)(size_t)(LDSP char*)smem + (ldw ? 8192 : 0) + lh * 4096;
        u32 vl[4];
#pragma unroll
        for (int i = 0; i < 4; ++i)
            vl[i] = ldw ? ((vw ^ (i << 4)) + i * (4 * WROW - 1024)) : (vx + i * (16 * XROW - 1024));
        const u32 vlin = lane * 16;
        const u32 locstep = ldw ? 32 * WROW : 64;
        const u32 lochalf = ldw ? 16 * WROW : 64 * XROW;
        const u32 kloc = is_p ? 0 : ga * 256;
        const u32 locS = (ldw ? ((1 - gc) * 2048 + o_cb * 128) * 2 : (rank_of(g) * 512 + o_ro * 128) * XROW) + lh * lochalf;
        const u32 locM = (ldw ? kloc * WROW + (gc * 2048 + m_cb * 128) * 2 : block_row(g, m_d, m_h) * XROW + kloc * 2)
                       + lh * lochalf - T0 * locstep;
        auto issue_loc = [&](u32 t, u32 half) {
            glds4(lds0 + (t & 7) * 16384 + half * 4096, vl[0], vl[1], vl[2], vl[3], srd_loc,
                  (t < T0 ? locS : locM) + t * locstep + half * lochalf);
        };
        volatile LDSP u32* full = ctl + (wave - 4);
        const u32 blk = ldw ? 16 + m_cb : m_d * 2 + m_h;
        const u32 stg0 = OFF_DATA + (call & 1) * PAR_BYTES + REL_IN + blk * PIECE + lh * 4096;
        const char* frow = myws + OFF_FIN + blk * 256 + lane * 4;
        auto stage = [&](auto halves) {
            constexpr int H = decltype(halves)::value, V = 4 * H;
            u32 pub = 0, lim = 8, t = 0;
            auto set_full = [&](u32 v) { full[0] = v; if (H == 2) full[2] = v; };
            auto publish1 = [&](auto extra) {
                constexpr int E = decltype(extra)::value;
                const u32 n = t - pub;
                if (n >= 6) wait_vm<5 * V + E>();
                else if (n == 5) wait_vm<4 * V + E>();
                else if (n == 4) wait_vm<3 * V + E>();
                else if (n == 3) wait_vm<2 * V + E>();
                else if (n == 2) wait_vm<V + E>();
                else wait_vm<E>();
                set_full(++pub);
            };
            using E0 = std::integral_constant<int, 0>;
            using E1 = std::integral_constant<int, 1>;
            auto make_room = [&]() {
                if (__builtin_expect(t - pub >= DEPTH, 1)) { wait_vm<V * (DEPTH - 1)>(); set_full(++pub); }
                while (__builtin_expect(t >= lim, 0)) {
                    const u32x4 dn = *(volatile LDSP u32x4*)(ctl + 4);
                    lim = uni(min(min(dn[0], dn[1]), min(dn[2], dn[3]))) + 8;
                    if (t < lim) break;
                    if (pub < t) publish1(E0{}); else __builtin_amdgcn_s_sleep(1);
                }
            };
            wait_vm<0>();
            for (; t < T1; ++t) {
                if (H == 2) __builtin_amdgcn_s_sleep(P_PACE);
                make_room();
                issue_loc(t, 0);
                if (H == 2) issue_loc(t, 1);
            }
            u64 mask = 0;
            u64 todo = 0xfefefefefefefefeull & ~(0x0101010101010101ull << (is_p ? 0 : m_d));
            bool slow = H == 2;
            do {
                if (slow) {
                    while (pub < t) publish1(E0{});
                    slow = uni(ctl[8] + ctl[9]) < 2;
                }
                make_room();
                const u32 b = __builtin_ctzll(todo);
                todo &= todo - 1;
                if (__builtin_expect(!((mask >> b) & 1), 0)) {
                    const u32 fv = ld_sys(frow);
                    while (pub < t) publish1(E1{});
                    mask = __ballot(fv >= call);
                    if (!((mask >> b) & 1)) mask = poll_mask(frow, 1ull << b, call, call, tmo);
                }
                glds4_stg(lds0 + (t & 7) * 16384, vlin, srd_ws, stg0 + b * 32 * PIECE);
                if (H == 2) glds4_stg(lds0 + (t & 7) * 16384 + 4096, vlin, srd_ws, stg0 + b * 32 * PIECE + 4096);
                ++t;
            } while (todo);
            while (pub < t) publish1(E0{});
            set_full(0x7fffffffu);
        };
        if (quiet) stage(std::integral_constant<int, 2>{});
        else stage(std::integral_constant<int, 1>{});
        return;
    }

    const u32 cbase = OFF_DATA + (call & 1) * PAR_BYTES + REL_CP;
    const u32 wm = wave & 1, wn = wave >> 1;
    const u32 g4 = lane >> 4, li = lane & 15;
    const u32 ax = wm * 4096 + li * 64 + ((g4 ^ ((0u - (li >> 2)) & 3)) << 4);
    u32 aw[4][2];
#pragma unroll
    for (int cc = 0; cc < 4; ++cc)
#pragma unroll
        for (int t2 = 0; t2 < 2; ++t2) {
            const u32 q = li >> 2, pp = lane & 3;
            aw[cc][t2] = 8192 + 256 * (8 * g4 + 4 * t2 + q)
                       + 16 * ((2 * (4 * wn + cc) + (pp >> 1)) ^ ((q << 2) | ((2 * g4 + t2) & 3))) + 8 * (pp & 1);
        }
    f32x4 acc[4][4], keep[4][4];
    auto readf = [&](u32 t, Frags& fr) {
        const LDSP char* sl = (const LDSP char*)smem + (t & 7) * 16384;
#pragma unroll
        for (int f = 0; f < 4; ++f) fr.x[f] = *(const LDSP bf16x8*)(sl + ax + f * 1024);
#pragma unroll
        for (int cc = 0; cc < 4; ++cc) {
            s16x4 lo = __builtin_amdgcn_ds_read_tr16_b64_v4i16((LDSP s16x4*)(sl + aw[cc][0]));
            s16x4 hi = __builtin_amdgcn_ds_read_tr16_b64_v4i16((LDSP s16x4*)(sl + aw[cc][1]));
            fr.w[cc] = __builtin_bit_cast(bf16x8, __builtin_shufflevector(lo, hi, 0, 1, 2, 3, 4, 5, 6, 7));
        }
    };
    auto mma = [&](const Frags& fr) {
#pragma unroll
        for (int c = 0; c < 4; ++c)
#pragma unroll
            for (int f = 0; f < 4; ++f)
                acc[c][f] = __builtin_amdgcn_mfma_f32_16x16x32_bf16(fr.w[c], fr.x[f], acc[c][f], 0, 0, 0);
    };
    auto interleave = [&]() {
#pragma unroll
        for (int i = 0; i < 4; ++i) {
            __builtin_amdgcn_sched_group_barrier(0x8, 1, 0);
            __builtin_amdgcn_sched_group_barrier(0x2, 2, 0);
            __builtin_amdgcn_sched_group_barrier(0x100, 2, 0);
        }
#pragma unroll
        for (int i = 0; i < 4; ++i) {
            __builtin_amdgcn_sched_group_barrier(0x8, 1, 0);
            __builtin_amdgcn_sched_group_barrier(0x2, 1, 0);
            __builtin_amdgcn_sched_group_barrier(0x100, 1, 0);
        }
        __builtin_amdgcn_sched_group_barrier(0x8, 8, 0);
        __builtin_amdgcn_sched_barrier(0);
    };
#pragma unroll
    for (int c = 0; c < 4; ++c)
#pragma unroll
        for (int f = 0; f < 4; ++f) acc[c][f] = f32x4{0.f, 0.f, 0.f, 0.f};
    u32 avail = 0;
    auto need = [&](u32 t) {
        while (__builtin_expect(avail <= t, 0)) {
            const u32x4 f = *(volatile LDSP u32x4*)ctl;
            avail = uni(min(min(f[0], f[1]), min(f[2], f[3])));
            if (avail <= t) __builtin_amdgcn_s_sleep(1);
        }
        asm volatile("" ::: "memory");
    };
    auto retire = [&](u32 t) {
        asm volatile("s_waitcnt lgkmcnt(0)" ::: "memory");
        ctl[4 + wave] = t + 1;
    };
    Frags fa, fb;
    need(0);
    readf(0, fa);
    auto steps = [&](u32 t0, u32 t1) {
        for (u32 t = t0; t < t1; t += 2) {
            retire(t);
            need(t + 1);
            readf(t + 1, fb);
            mma(fa);
            interleave();
            retire(t + 1);
            need(t + 2);
            readf(t + 2, fa);
            mma(fb);
            interleave();
        }
    };
    steps(0, T0);
#pragma unroll
    for (int c = 0; c < 4; ++c)
#pragma unroll
        for (int f = 0; f < 4; ++f) { keep[c][f] = acc[c][f]; acc[c][f] = f32x4{0.f, 0.f, 0.f, 0.f}; }
    STAMP(2);
    steps(T0, T);
    STAMP(3);

    if (!is_p) {
        char* dws = p.peer[m_d];
        auto rd = __builtin_amdgcn_make_buffer_rsrc((void*)dws, 0, WS_NEED, 0x00020000);
        const u32 ci = (m_d * 32 + m_h * 16 + m_cb) * 4 + wave;
        u32x4 pk[8];
#pragma unroll
        for (int f = 0; f < 4; ++f)
#pragma unroll
            for (int P = 0; P < 2; ++P) {
                const f32x4 lo = acc[2 * P][f], hi = acc[2 * P + 1][f];
                pk[f * 2 + P] = u32x4{pack2(lo[0], lo[1]), pack2(lo[2], lo[3]), pack2(hi[0], hi[1]), pack2(hi[2], hi[3])};
            }
        char* tok = myws + OFF_TOK + m_d * 256;
        const u32 pos = (role >> 8) & 31;
        if (wave == 0)
            poll_mask(tok, 1ull, (call - 1) * (NTOK + 32 * C_SHARE) + max((pos + 1) * C_SHARE, (u32)G_MIN), call, tmo);
        asm volatile("s_waitcnt vmcnt(0)\n\ts_barrier" ::: "memory");
        STAMP(7);
        const bool wide = pos >= C_LAST_POS;
#pragma unroll
        for (int u = 0; u < 8; ++u) {
            if (wide) wait_vm<C_WIN_LAST - 1>(); else wait_vm<C_WIN - 1>();
            __builtin_amdgcn_raw_buffer_store_b128(pk[u], rd, lane * 16 + u * 1024, cbase + ci * PIECE, AUX_SYS);
            if (u == C_REL && wave == 0 && lane == 0) add_agent(tok, C_SHARE);
        }
        wait_vm<0>();
        if (lane == 0) st_sys(dws + OFF_FC + ci * 64, call);
    } else {
#pragma unroll
        for (int c = 0; c < 4; ++c)
#pragma unroll
            for (int f = 0; f < 4; ++f) keep[c][f] = acc[c][f];
    }
    STAMP(4);

    if (is_p || has_small) {
        const u32 db = ((o_ro >> 1) ^ gb) << 1;
        const u32 l0 = is_p ? (1 | db) : (4 | db);
        const u32 tile = (o_ro & 1) * 16 + o_cb;
        auto rs = __builtin_amdgcn_make_buffer_rsrc((void*)myws, 0, WS_NEED, 0x00020000);
        const u32 ci0 = (l0 * 32 + tile) * 4 + wave;
        u32 left = is_p ? 1 : 3, tmo_v = 0;
        while (left) {
            const u32 got = (u32)poll_any(myws + OFF_FC + (ci0 + (lane & 1) * 128) * 64, left, call, tmo) & left;
            const u32 k = (got & 1) ? 0 : 1;
            left &= ~(1u << k);
            u32x4 v[8];
#pragma unroll
            for (int u = 0; u < 8; ++u)
                v[u] = __builtin_amdgcn_raw_buffer_load_b128(rs, lane * 16 + u * 1024, cbase + (ci0 + k * 128) * PIECE, STG_AUXN);
            tmo_v = ld_agent(tmo);
#pragma unroll
            for (int f = 0; f < 4; ++f)
#pragma unroll
                for (int P = 0; P < 2; ++P) {
                    const u32x4 t = v[f * 2 + P];
                    keep[2 * P][f] += f32x4{bf_lo(t[0]), bf_hi(t[0]), bf_lo(t[1]), bf_hi(t[1])};
                    keep[2 * P + 1][f] += f32x4{bf_lo(t[2]), bf_hi(t[2]), bf_lo(t[3]), bf_hi(t[3])};
                }
        }
        STAMP(5);
        const bool bad = uni(tmo_v) == call;
        auto ro = __builtin_amdgcn_make_buffer_rsrc((void*)p.out, 0, IN_BYTES, 0x00020000);
        const u32 ocol = (is_p ? gc : 1 - gc) * 2048 + o_cb * 128 + wn * 64;
        const u32 obase = (o_ro * 128 + wm * 64) * WROW + ocol * 2;
        const u32 vo = li * WROW + (g4 & 1) * 32 + (g4 >> 1) * 16;
#pragma unroll
        for (int f = 0; f < 4; ++f)
#pragma unroll
            for (int P = 0; P < 2; ++P) {
                const f32x4 lo = keep[2 * P][f], hi = keep[2 * P + 1][f];
                auto r0 = __builtin_amdgcn_permlane16_swap(pack2(lo[0], lo[1]), pack2(hi[0], hi[1]), false, false);
                auto r1 = __builtin_amdgcn_permlane16_swap(pack2(lo[2], lo[3]), pack2(hi[2], hi[3]), false, false);
                u32x4 o = u32x4{r0[0], r1[0], r0[1], r1[1]};
                if (bad) o = u32x4{0x7fc07fc0u, 0x7fc07fc0u, 0x7fc07fc0u, 0x7fc07fc0u};
                __builtin_amdgcn_raw_buffer_store_b128(o, ro, vo + P * 64, obase + f * 16 * WROW, 0);
            }
    }
    if (threadIdx.x == 0) st_agent(ctr, call);
    STAMP(6);
}

static void build_schedule(Params& p) {
    static const int CL[6] = {1, 3, 4, 5, 6, 7};
    int nsmall = 0;
    for (int n = 0; n < 64; ++n)
        p.role[(n & 31) * 8 + (n >> 5) * 2] = (u32)(((n >> 1) & 1) * 2 | (n & 1) << 3 | (n >> 2) << 4);
    for (int pos = 0; pos < 32; ++pos)
        for (int li = 0; li < 6; ++li) {
            u32 e = (u32)(CL[li] | (pos & 1) << 3 | (pos >> 1) << 4 | pos << 8);
            if (nsmall < 64) { e |= 1u << 13 | (u32)nsmall << 14; ++nsmall; }
            p.role[((pos + 8) & 31) * 8 + CL[li]] = e;
        }
    for (int l = 0; l < 8; ++l) {
        unsigned char lst[256] = {0};
        int n = 0;
        if (l) {
            for (int r = 0; r < 8 + LAG; ++r) {
                if (r < 8) {
                    for (int li = 0; li < 6; ++li)
                        if (CL[li] != l) { lst[n++] = (unsigned char)(CL[li] * 2 | r << 5); lst[n++] = (unsigned char)((CL[li] * 2 + 1) | r << 5); }
                    for (int c = 0; c < 8; ++c) lst[n++] = (unsigned char)((16 + c) | r << 5);
                }
                if (r >= OWN_LAG && r < 8 + OWN_LAG)
                    for (int k = 0; k < 4; ++k) lst[n++] = (unsigned char)(((k >> 1) * 4 + (k & 1)) | (r - OWN_LAG) << 5);
                if (r >= LAG)
                    for (int c = 8; c < 16; ++c) lst[n++] = (unsigned char)((16 + c) | (r - LAG) << 5);
            }
        }
        for (int w = 0; w < (int)NTOK; ++w) {
            u64 v[2] = {0, 0};
            for (int k = 0; k < 16; ++k) v[k >> 3] |= (u64)lst[w + 16 * k] << (8 * (k & 7));
            p.plist[l][w][0] = v[0]; p.plist[l][w][1] = v[1];
        }
    }
}

extern "C" void kernel_launch(int rank, int nranks, ncclComm_t comm,
                              hipStream_t stream,
                              void* const* d_in, const int* in_sizes, int n_in,
                              void* d_out, int out_size,
                              const int* in_dtypes, int out_dtype,
                              void* const* d_out_all,
                              void* const* d_ws_all, size_t ws_bytes) {
    if (nranks != NR || (unsigned)rank >= (unsigned)NR || n_in < 2 || ws_bytes < WS_NEED ||
        in_sizes[0] != 4096 * 512 || in_sizes[1] != 512 * 4096 || out_size != 512 * 4096) {
        fprintf(stderr, "kernel_launch: unexpected problem shape\n");
        return;
    }
    Params p;
    p.x = (const char*)d_in[0];
    p.w = (const char*)d_in[1];
    p.out = (char*)d_out;
    u32 id = 0;
    while (rank_of(id) != (u32)rank) ++id;
    for (u32 d = 0; d < NR; ++d) p.peer[d] = (char*)d_ws_all[rank_of(id ^ d)];
    p.self = p.peer[0];
    p.rank = id;
    p.pad = 0;
    build_schedule(p);
    hipLaunchKernelGGL(gemm_rs_kernel, dim3(256), dim3(512), 0, stream, p);
    HIP_CHECK(hipGetLastError());
}
```

```cpp
#include <hip/hip_runtime.h>
#include <rccl/rccl.h>
#include <cstdio>
#include <type_traits>

#define HIP_CHECK(expr) do { hipError_t _e = (expr); if (_e != hipSuccess) { \
    fprintf(stderr, "HIP error %s at %s:%d\n", hipGetErrorString(_e), __FILE__, __LINE__); \
    return; } } while (0)

typedef unsigned u32;
typedef unsigned long long u64;
typedef u32 u32x4 __attribute__((ext_vector_type(4)));
typedef short s16x4 __attribute__((ext_vector_type(4)));
typedef __bf16 bf16x8 __attribute__((ext_vector_type(8)));
typedef __bf16 bf16x2 __attribute__((ext_vector_type(2)));
typedef float f32x4 __attribute__((ext_vector_type(4)));
typedef float f32x2 __attribute__((ext_vector_type(2)));
#define LDSP __attribute__((address_space(3)))
#define GLBP __attribute__((address_space(1)))

constexpr int NR = 8;
constexpr u32 IN_BYTES = 4u << 20;
constexpr u32 XROW = 1024;
constexpr u32 WROW = 8192;
constexpr u32 PIECE = 8192;
constexpr u32 OFF_FIN = 0;
constexpr u32 OFF_LF = 65536;
constexpr u32 OFF_CTR = 131072;
constexpr u32 OFF_CLAIM = 163840;
constexpr u32 OFF_TMO = 196608;
constexpr u32 OFF_TOK = 200704;
constexpr u32 OFF_DATA = 2u << 20;
constexpr u32 PAR_BYTES = 24u << 20;
constexpr u32 REL_IN = 0;
constexpr u32 REL_CP = 16u << 20;
constexpr u32 WS_NEED = OFF_DATA + 2 * PAR_BYTES;
constexpr u64 DEADLINE_TICKS = 50000000ull;
constexpr int AUX_SYS = 17;
constexpr int AUX_OUT = 16;
constexpr int PUSH_WIN = 7;
constexpr int PULL_AT = 3;
constexpr int C_WIN = 2;
constexpr int C_REL = 5;
constexpr u32 PRZ = 4;
constexpr int FWD = 6, FWD1 = 4, FWD3 = 20;
constexpr int DEPTH = 4;
constexpr int P_PACE = 32, R_PACE = 16;
constexpr int POLL_SLEEP = 2;
constexpr u32 EARLY = 4;
constexpr u32 NTOK = 16;
constexpr u32 QLEN = 24;
constexpr u32 C_SHARE = 2;

struct Params {
    const char* x;
    const char* w;
    char* out;
    char* self;
    char* peer[NR];
    u32 rank;
    u32 pad;
    u32 npieces[NR];
    u32 role[256];
    u64 plist[NR][8][4];
    unsigned char order[12][64];
};

template <int N> __device__ __forceinline__ void wait_vm() {
    asm volatile("s_waitcnt vmcnt(%0)" :: "i"(N) : "memory");
}
__device__ __forceinline__ u32 uni(u32 v) { return __builtin_amdgcn_readfirstlane(v); }
__device__ __forceinline__ u32x4 make_srd(const void* p, u32 bytes) {
    u64 a = (u64)p;
    u32x4 r;
    r[0] = uni((u32)a); r[1] = uni((u32)(a >> 32) & 0xffffu); r[2] = bytes; r[3] = 0x00020000u;
    return r;
}
#define GLDS4(POLICY)                                                                  \
    asm volatile("s_mov_b32 m0, %0\n\t"                                                \
                 "s_nop 0\n\t"                                                         \
                 "buffer_load_dwordx4 %1, %5, %6 offen " POLICY " lds\n\t"             \
                 "buffer_load_dwordx4 %2, %5, %6 offen offset:1024 " POLICY " lds\n\t" \
                 "buffer_load_dwordx4 %3, %5, %6 offen offset:2048 " POLICY " lds\n\t" \
                 "buffer_load_dwordx4 %4, %5, %6 offen offset:3072 " POLICY " lds"     \
                 :: "s"(dst), "v"(v0), "v"(v1), "v"(v2), "v"(v3), "s"(srd), "s"(soff)  \
                 : "memory")
__device__ __forceinline__ void glds4(u32 dst, u32 v0, u32 v1, u32 v2, u32 v3, u32x4 srd, u32 soff) {
    GLDS4("");
}
__device__ __forceinline__ void glds4_stg(u32 dst, u32 v0, u32x4 srd, u32 soff) {
    const u32 v1 = v0, v2 = v0, v3 = v0;
    GLDS4("sc0 sc1");
}
__device__ __forceinline__ u32 ld_sys(const char* p) {
    return __hip_atomic_load((const GLBP u32*)p, __ATOMIC_RELAXED, __HIP_MEMORY_SCOPE_SYSTEM);
}
__device__ __forceinline__ void st_sys(char* p, u32 v) {
    __hip_atomic_store((GLBP u32*)p, v, __ATOMIC_RELAXED, __HIP_MEMORY_SCOPE_SYSTEM);
}
__device__ __forceinline__ u32 ld_agent(const char* p) {
    return __hip_atomic_load((const GLBP u32*)p, __ATOMIC_RELAXED, __HIP_MEMORY_SCOPE_AGENT);
}
__device__ __forceinline__ void st_agent(char* p, u32 v) {
    __hip_atomic_store((GLBP u32*)p, v, __ATOMIC_RELAXED, __HIP_MEMORY_SCOPE_AGENT);
}
__device__ __forceinline__ void add_agent(char* p, u32 v) {
    __hip_atomic_fetch_add((GLBP u32*)p, v, __ATOMIC_RELAXED, __HIP_MEMORY_SCOPE_AGENT);
}
__device__ __forceinline__ u64 poll_mask(const char* p, u64 need, u32 thr, u32 call, char* tmo, volatile LDSP u32* lost) {
    u64 t0 = 0;
    for (u32 spin = 0;; ++spin) {
        const u64 m = __ballot(ld_sys(p) >= thr);
        if (m & need) return m;
        if (spin == 0) t0 = __builtin_amdgcn_s_memrealtime();
        if ((spin & 15u) == 15u) {
            if (uni(ld_agent(tmo)) == call || __builtin_amdgcn_s_memrealtime() - t0 > DEADLINE_TICKS) {
                st_agent(tmo, call);
                *lost = 1;
                return ~0ull;
            }
        }
        __builtin_amdgcn_s_sleep(POLL_SLEEP);
    }
}
__device__ __forceinline__ u32 pack2(float lo, float hi) {
    return __builtin_bit_cast(u32, __builtin_convertvector(f32x2{lo, hi}, bf16x2));
}
__device__ __forceinline__ u32 pack2nz(float lo, float hi) {
    const u32 v = pack2(lo, hi);
    return v ? v : 0x80008000u;
}
__device__ __forceinline__ float bf_lo(u32 v) { return __builtin_bit_cast(float, v << 16); }
__device__ __forceinline__ float bf_hi(u32 v) { return __builtin_bit_cast(float, v & 0xffff0000u); }
constexpr u32 RANK_OF = 0x75416320u;
__host__ __device__ __forceinline__ u32 rank_of(u32 id) { return (RANK_OF >> (4 * id)) & 7u; }
__host__ __device__ __forceinline__ u32 col_class(u32 cb) { return (cb >> 2) + (cb >= 14); }
__device__ __forceinline__ u32 block_row(u32 r, u32 rel, u32 h) {
    const u32 own2 = rel == 2;
    return rank_of(own2 ? r : r ^ rel) * 512 + (((r >> 1) & 1) ^ own2) * 256 + h * 128;
}

struct Frags { bf16x8 x[4]; bf16x8 w[4]; };
constexpr u32 LDS_PSH = 8 * 16384;
constexpr u32 LDS_CTL = LDS_PSH + 3 * PIECE;

__global__ void __launch_bounds__(512, 2) gemm_rs_kernel(Params p) {
    __shared__ __attribute__((aligned(16))) char smem[LDS_CTL + 128];
    const u32 lane = threadIdx.x & 63;
    const u32 wave = uni(threadIdx.x >> 6);
    u32 item = uni((blockIdx.x & ~7u) | (__builtin_amdgcn_s_getreg((3 << 11) | 20) & 7u));
    u32 vx = (lane >> 2) * XROW + (((lane & 3) ^ ((0u - (lane >> 4)) & 3)) << 4);
    u32 vw = (lane >> 4) * WROW + (((lane & 15) ^ ((lane >> 4) << 2)) << 4);
    const char* px = p.x;
    const char* pw = p.w;
    char* myws = p.self;
    u32 g = p.rank;
    u64 lst0 = p.plist[item & 7][(item >> 3) & 7][0], lst1 = p.plist[item & 7][(item >> 3) & 7][1];
    u64 lst2 = p.plist[item & 7][(item >> 3) & 7][2], lst3 = p.plist[item & 7][(item >> 3) & 7][3];
    char* lws = p.peer[item & 7];
    u32 npc = p.npieces[item & 7];
    asm volatile("" : "+s"(px), "+s"(pw), "+s"(myws), "+s"(g), "+s"(lst0), "+s"(lst1), "+s"(lst2), "+s"(lst3), "+s"(lws), "+s"(npc),
                      "+v"(vx), "+v"(vw));
    g &= 7u;
    const u32 ga = g & 1, gb = (g >> 1) & 1, gc = g >> 2;
    char* tmo = myws + OFF_TMO;
    char* ctr = myws + OFF_CTR + blockIdx.x * 128;
    volatile LDSP u32* ctl = (volatile LDSP u32*)(smem + LDS_CTL);
    if (wave == 0) {
        if (lane == 0) {
            GLBP u32* claim = (GLBP u32*)(myws + OFF_CLAIM);
            const u32 old = __hip_atomic_fetch_add(claim + item * 32, 1u, __ATOMIC_RELAXED, __HIP_MEMORY_SCOPE_AGENT);
            const u32 calls = ld_agent(ctr);
            u32 it = item;
            if (__builtin_expect(old != calls, 0)) {
                __hip_atomic_fetch_add(claim + item * 32, ~0u, __ATOMIC_RELAXED, __HIP_MEMORY_SCOPE_AGENT);
#pragma clang loop unroll(disable)
                for (u32 tries = 0;; ++tries) {
                    it = (it + 1) & 255u;
                    u32 expect = calls;
                    if (__hip_atomic_compare_exchange_strong(claim + it * 32, &expect, calls + 1, __ATOMIC_RELAXED,
                                                             __ATOMIC_RELAXED, __HIP_MEMORY_SCOPE_AGENT)) break;
                    if (tries > 4096) { st_agent(tmo, calls + 1); break; }
                }
            }
            ctl[10] = it; ctl[11] = calls + 1;
        } else if (lane <= 14) {
            ctl[lane >= 11 ? lane + 1 : lane - 1] = 0;
        }
    }
    auto pushes = [](u32 it) { return (it & 7) != 0 && it < 64; };
    const u32x4 sx = make_srd(px, IN_BYTES), sw = make_srd(pw, IN_BYTES);
    auto fetch_own = [&](u32 e, u32 dst, u32 slot) {
        const u32 blk = e & 31, k0 = (dst & 1) * 256 + (e >> 5) * 32;
        const u32 ld = (u32)(size_t)(LDSP char*)smem + LDS_PSH + slot * PIECE;
        if (blk >= 16) {
            const u32 base = k0 * WROW + ((dst >> 2) * 2048 + (blk - 16) * 128) * 2;
            constexpr u32 D = 4 * WROW - 1024;
            glds4(ld, vw, (vw ^ 16) + D, (vw ^ 32) + 2 * D, (vw ^ 48) + 3 * D, sw, base);
            glds4(ld + 4096, vw, (vw ^ 16) + D, (vw ^ 32) + 2 * D, (vw ^ 48) + 3 * D, sw, base + 16 * WROW);
        } else {
            const u32 base = block_row(dst, blk >> 1, blk & 1) * XROW + k0 * 2;
            constexpr u32 D = 16 * XROW - 1024;
            glds4(ld, vx, vx + D, vx + 2 * D, vx + 3 * D, sx, base);
            glds4(ld + 4096, vx, vx + D, vx + 2 * D, vx + 3 * D, sx, base + 64 * XROW);
        }
    };
    auto head_start = [&]() {
#pragma clang loop unroll(disable)
        for (u32 i = 7 - wave; i < 15 - 2 * wave; ++i) fetch_own((u32)(lst0 >> (8 * i)) & 255u, g ^ (item & 7), i);
    };
    if (wave >= 6 && pushes(item)) head_start();
    if (wave == 5 && pushes(item) && lane < 2) {
        const u32 o = OFF_DATA + lane * PAR_BYTES + REL_IN + (item & 7) * 256 * PIECE;
        asm volatile("s_mov_b32 m0, %0\n\ts_nop 0\n\tglobal_load_lds_dword %1, %2 sc0 sc1"
                     :: "s"((u32)(size_t)(LDSP char*)smem + LDS_CTL + 64), "v"(o), "s"(lws) : "memory");
    }
    asm volatile("s_waitcnt lgkmcnt(0)\n\ts_barrier" ::: "memory");
    const u32 call = uni(ctl[11]);
    {
        const u32 it = uni(ctl[10]);
        if (__builtin_expect(it != item, 0)) {
            item = it;
            lst0 = p.plist[item & 7][(item >> 3) & 7][0]; lst1 = p.plist[item & 7][(item >> 3) & 7][1];
            lst2 = p.plist[item & 7][(item >> 3) & 7][2]; lst3 = p.plist[item & 7][(item >> 3) & 7][3];
            lws = p.peer[item & 7];
            npc = p.npieces[item & 7];
            wait_vm<0>();
            if (wave >= 6 && pushes(item)) head_start();
        }
    }
    if (wave == 7 && pushes(item)) {
        auto r0 = __builtin_amdgcn_make_buffer_rsrc((void*)lws, 0, WS_NEED, 0x00020000);
        const u32 o = OFF_DATA + (call & 1) * PAR_BYTES + REL_IN + ((item & 7) * 256 + ((u32)lst0 & 255u)) * PIECE;
        u32x4 v[8];
        wait_vm<0>();
#pragma unroll
        for (int u = 0; u < 8; ++u) v[u] = *(const LDSP u32x4*)(smem + LDS_PSH + u * 1024 + lane * 16);
#pragma unroll
        for (int u = 0; u < 8; ++u) __builtin_amdgcn_raw_buffer_store_b128(v[u], r0, lane * 16 + u * 1024, o, AUX_SYS);
        asm volatile("" ::: "memory");
        ctl[15] = 1;
    }
    const bool passes = (item & 7) == 0 && item < 64;
    const bool quiet = pushes(item) || passes;

    if (__builtin_expect(quiet && wave >= 6, 1)) {
        const u32 link = passes ? 2 : item & 7;
        const u32 nx = link == 2 ? 0 : link == 1 ? 16 * PRZ + 32 : link == 3 ? 32 : 32 - 4 * PRZ;
        const u32 nl = (npc + 7 - (item >> 3)) >> 3, n = (npc + nx + 7 - (item >> 3)) >> 3;
        auto extra = [&](u32 k, u32& src, u32& dst) {
            const u32 x = (item >> 3) + 8 * k - npc, ti = x >> 2;
            const bool gift = link == 1 && ti < 4 * PRZ;
            const u32 rel = gift ? 4 + (ti & 3) : link;
            const u32 cb = gift ? ti >> 2 : link == 1 ? ti - 4 * PRZ : link == 3 ? ti : ti + PRZ;
            src = (rel * 8 + cb) * 4 + (x & 3);
            dst = (gift ? ((rel ^ 1) - 4) * 8 + cb : rel * 32 + ga * 16 + cb) * 4 + (x & 3);
        };
        const u32 pb = OFF_DATA + (call & 1) * PAR_BYTES + REL_IN;
        const u32 bits = passes ? 16 : 8, emask = passes ? 0x7ffu : 0xffu;
        auto next = [&]() {
            const u32 e = (u32)lst0 & emask;
            lst0 = (lst0 >> bits) | (lst1 << (64 - bits)); lst1 = (lst1 >> bits) | (lst2 << (64 - bits));
            lst2 = (lst2 >> bits) | (lst3 << (64 - bits)); lst3 >>= bits;
            return e;
        };
        if (wave == 6) {
            const u32x4 ss = make_srd(myws, WS_NEED);
            if (passes) {
                const u64 lw = (lane & 8) ? ((lane & 4) ? lst3 : lst2) : ((lane & 4) ? lst1 : lst0);
                const u32 le = (u32)(lw >> (16 * (lane & 3))) & 0x7ffu;
                const char* fl = myws + OFF_FIN + ((le & 31) * 64 + (le >> 5)) * 4;
                for (u64 want = __ballot(le >= 256) & 0xffff; want;) want &= ~poll_mask(fl, want, call, call, tmo, ctl + 12);
                wait_vm<0>();
            } else {
                wait_vm<8>(); ctl[14] = 2;
                next(); next(); next();
            }
            u32 slot = 0, taken = 0, le, ld2;
            extra(min(nl + (lane & 15), max(n, nl + 1) - 1), le, ld2);
            const char* lf = myws + OFF_LF + (le & 255) * 64;
            u64 parked = 0;
            for (u32 k = passes ? 0 : 3; k < n; ++k) {
                if (k >= taken + 3) {
                    wait_vm<0>();
                    ctl[14] = k;
                    while (k >= (taken = uni(ctl[15])) + 3) __builtin_amdgcn_s_sleep(1);
                }
                u32 e = next();
                const u32 ld = (u32)(size_t)(LDSP char*)smem + LDS_PSH + slot * PIECE;
                if (k >= nl) {
                    u32 d2;
                    extra(k, e, d2);
                    if (!((parked >> (k - nl)) & 1)) {
                        wait_vm<0>();
                        ctl[14] = k;
                        parked = poll_mask(lf, 1ull << (k - nl), call, call, tmo, ctl + 12);
                    }
                }
                if (e >= 256 || k >= nl) {
                    glds4_stg(ld, lane * 16, ss, pb + e * PIECE);
                    glds4_stg(ld + 4096, lane * 16, ss, pb + e * PIECE + 4096);
                } else {
                    fetch_own(e, g ^ link, slot);
                }
                slot = slot == 2 ? 0 : slot + 1;
                wait_vm<8>();
                ctl[14] = k;
            }
            wait_vm<0>();
            ctl[14] = n;
            return;
        }
        char* dws = passes ? p.peer[2] : lws;
        auto rd = __builtin_amdgcn_make_buffer_rsrc((void*)dws, 0, WS_NEED, 0x00020000);
        u32x4 cur[8], nxt[8];
        const u32 k0 = passes ? 0 : 1;
        u32 slot = k0, got = 0, lost = 0;
        auto landed = [&](u32 k) { return got > k || (got = uni(ctl[14])) > k; };
        auto pull = [&](u32x4 (&v)[8], u32 k) {
            asm volatile("" ::: "memory");
#pragma unroll
            for (int u = 0; u < 8; ++u) v[u] = *(const LDSP u32x4*)(smem + LDS_PSH + slot * PIECE + u * 1024 + lane * 16);
            const u32 lw = ctl[12];
            asm volatile("s_waitcnt lgkmcnt(0)" ::: "memory");
            ctl[15] = k + 1;
            lost = uni(lw);
            slot = slot == 2 ? 0 : slot + 1;
        };
        u32 prev_flag = ~0u;
        if (!passes) {
            const u32 e = next() | link << 8;
            prev_flag = OFF_FIN + ((e & 31) * 64 + (e >> 5)) * 4;
        }
        if (n) {
            while (!landed(k0)) __builtin_amdgcn_s_sleep(1);
            pull(cur, k0);
        }
        if (passes) poll_mask(myws + OFF_TOK + 2 * 256, 1ull, (call - 1) * NTOK + 2 * (item >> 3) + 2, call, tmo, ctl + 12);
        for (u32 k = k0; k < n; ++k) {
            if (lost) { ctl[15] = n; break; }
            u32 e = next() ^ (link << 8);
            if (k >= nl) {
                u32 s2;
                extra(k, s2, e);
                e += (REL_CP - REL_IN) / PIECE;
            }
            bool have = k + 1 == n;
#pragma unroll
            for (int u = 0; u < 8; ++u) {
                wait_vm<PUSH_WIN - 1>();
                if (u == PUSH_WIN - 1 && prev_flag != ~0u && lane == 0) st_sys(dws + prev_flag, call);
                __builtin_amdgcn_raw_buffer_store_b128(cur[u], rd, lane * 16 + u * 1024, pb + e * PIECE, AUX_SYS);
                if (u == PULL_AT && !have && landed(k + 1)) { pull(nxt, k + 1); have = true; }
            }
            prev_flag = k < nl ? OFF_FIN + ((e & 31) * 64 + (e >> 5)) * 4 : ~0u;
            if (!have) {
                if (!landed(k + 1)) {
                    wait_vm<0>();
                    if (prev_flag != ~0u && lane == 0) st_sys(dws + prev_flag, call);
                    prev_flag = ~0u;
                    while (!landed(k + 1)) __builtin_amdgcn_s_sleep(1);
                }
                pull(nxt, k + 1);
            }
#pragma unroll
            for (int u = 0; u < 8; ++u) cur[u] = nxt[u];
        }
        if (!passes && lane == 0) add_agent(myws + OFF_TOK + link * 256, 2);
        ctl[8] = 2;
        wait_vm<0>();
        if (prev_flag != ~0u && lane == 0) st_sys(dws + prev_flag, call);
        return;
    }

    const u32 role = p.role[item];
    const u32 m_d = role & 7, m_h = (role >> 3) & 1, m_cb = (role >> 4) & 15;
    const bool is_p = (m_d & 5) == 0;
    const bool has_small = (role >> 14) & 1;
    const u32 o_ro = is_p ? ((gb ^ (m_d >> 1)) << 1) | m_h : (role >> 15) & 3;
    const u32 o_cb = is_p ? m_cb : (role >> 17) & 15;
    const bool twice = (role >> 30) & 1;
    const bool nomain = role >> 31;
    const u32 a_cb = (role >> 15) & 15;
    const u32 skip = (role >> 21) & 255;
    const bool whole = (role >> 29) & 1;
    const u32 nloc = whole ? 16 : 8;
    const u32 Tm = nloc + 56 - 8 * __builtin_popcount(skip);
    const u32 T0 = has_small ? 16 : twice ? Tm : 0;
    const u32 T = T0 + (nomain ? 0 : Tm);
    const bool sends = !is_p && !nomain;
    char* tok = myws + OFF_TOK + m_d * 256;
    const u32 pos = (role >> 8) & 63;

    if (wave >= 4) {
        const bool ldw = wave & 1;
        const u32 lh = (wave >> 1) & 1;
        const u32x4 srd_loc = ldw ? sw : sx;
        const u32x4 srd_ws = make_srd(myws, WS_NEED);
        asm volatile("s_nop 4" :: "s"(srd_loc), "s"(srd_ws));
        const u32 lds0 = (u32)(size_t)(LDSP char*)smem + (ldw ? 8192 : 0) + lh * 4096;
        u32 vl[4];
#pragma unroll
        for (int i = 0; i < 4; ++i)
            vl[i] = ldw ? ((vw ^ (i << 4)) + i * (4 * WROW - 1024)) : (vx + i * (16 * XROW - 1024));
        const u32 vlin = lane * 16;
        const u32 locstep = ldw ? 32 * WROW : 64;
        const u32 lochalf = ldw ? 16 * WROW : 64 * XROW;
        const u32 kloc = whole ? 0 : ga * 256;
        volatile LDSP u32* full = ctl + (wave - 4);
        u32 order0 = p.order[10][lane];
        u32 order1 = p.order[is_p && quiet ? 11 : (is_p ? 5 : 0) + col_class(m_cb)][lane];
        asm volatile("" : "+v"(order0), "+v"(order1));
        auto stage = [&](auto halves) {
            constexpr int H = decltype(halves)::value, V = 4 * H;
            u32 pub = 0, lim = 8, t = 0;
            auto set_full = [&](u32 v) { full[0] = v; if (H == 2) full[2] = v; };
            auto publish1 = [&](auto extra) {
                constexpr int E = decltype(extra)::value;
                const u32 n = t - pub;
                if (n >= 6) wait_vm<5 * V + E>();
                else if (n == 5) wait_vm<4 * V + E>();
                else if (n == 4) wait_vm<3 * V + E>();
                else if (n == 3) wait_vm<2 * V + E>();
                else if (n == 2) wait_vm<V + E>();
                else wait_vm<E>();
                set_full(++pub);
            };
            using E0 = std::integral_constant<int, 0>;
            using E1 = std::integral_constant<int, 1>;
            auto make_room = [&]() {
                if (__builtin_expect(t - pub >= DEPTH, 1)) { wait_vm<V * (DEPTH - 1)>(); set_full(++pub); }
                while (__builtin_expect(t >= lim, 0)) {
                    const u32x4 dn = *(volatile LDSP u32x4*)(ctl + 4);
                    lim = uni(min(min(dn[0], dn[1]), min(dn[2], dn[3]))) + 8;
                    if (t < lim) break;
                    if (pub < t) publish1(E0{}); else __builtin_amdgcn_s_sleep(1);
                }
            };
            wait_vm<0>();
            bool slow = H == 2 && !passes;
            if (nomain) while (slow) { __builtin_amdgcn_s_sleep(8); slow = uni(ctl[8]) < 2; }
            auto throttle = [&](auto pace) {
                if (slow) {
                    __builtin_amdgcn_s_sleep(decltype(pace)::value);
                    slow = uni(ctl[8]) < 2;
                }
            };
            for (u32 seg = T0 ? 0 : 1; seg < (nomain ? 1u : 2u); ++seg) {
                const bool sm = seg == 0 && has_small;
                const u32 h = seg ? m_h : ga, cb = seg ? m_cb : a_cb;
                u32 loc = sm ? (ldw ? ((1 - gc) * 2048 + o_cb * 128) * 2 : (rank_of(g) * 512 + o_ro * 128) * XROW)
                             : (ldw ? kloc * WROW + (gc * 2048 + cb * 128) * 2 : block_row(g, m_d, h) * XROW + kloc * 2);
                loc += lh * lochalf;
                for (u32 k = sm ? 16 : nloc; k; --k, ++t, loc += locstep) {
                    throttle(std::integral_constant<int, P_PACE>{});
                    make_room();
                    glds4(lds0 + (t & 7) * 16384, vl[0], vl[1], vl[2], vl[3], srd_loc, loc);
                    if (H == 2) glds4(lds0 + (t & 7) * 16384 + 4096, vl[0], vl[1], vl[2], vl[3], srd_loc, loc + lochalf);
                }
                if (sm) continue;
                const u32 blk = ldw ? 16 + cb : m_d * 2 + h;
                const u32 stg0 = OFF_DATA + (call & 1) * PAR_BYTES + REL_IN + blk * PIECE + lh * 4096;
                const char* frow = myws + OFF_FIN + blk * 256 + lane * 4;
                const u32 order = seg ? order1 : order0;
                u64 mask = 0;
                for (u32 i = 0; i < 56; ++i) {
                    const u32 b = __builtin_amdgcn_readlane(order, i);
                    if ((skip >> (b >> 3)) & 1) continue;
                    throttle(std::integral_constant<int, R_PACE>{});
                    make_room();
                    if (__builtin_expect(!((mask >> b) & 1), 0)) {
                        const u32 fv = ld_sys(frow);
                        while (pub < t) publish1(E1{});
                        mask = __ballot(fv >= call);
                        if (!((mask >> b) & 1)) mask = poll_mask(frow, 1ull << b, call, call, tmo, ctl + 12);
                    }
                    glds4_stg(lds0 + (t & 7) * 16384, vlin, srd_ws, stg0 + b * 32 * PIECE);
                    if (H == 2) glds4_stg(lds0 + (t & 7) * 16384 + 4096, vlin, srd_ws, stg0 + b * 32 * PIECE + 4096);
                    ++t;
                }
            }
            while (pub < t) publish1(E0{});
            set_full(0x7fffffffu);
        };
        if (quiet) stage(std::integral_constant<int, 2>{});
        else stage(std::integral_constant<int, 1>{});
        if (wave == 4 && sends) {
            poll_mask(tok, 1ull, (call - 1) * (NTOK + QLEN * C_SHARE) + (pos + 1) * C_SHARE, call, tmo, ctl + 12);
            ctl[13] = 1;
        }
        return;
    }

    const u32 cbase = OFF_DATA + (call & 1) * PAR_BYTES + REL_CP;
    char* cws = p.peer[m_d];
    char* pout = p.out;
    asm volatile("" : "+s"(cws), "+s"(pout));
    const u32 wm = wave & 1, wn = wave >> 1;
    const u32 g4 = lane >> 4, li = lane & 15;
    const u32 ax = wm * 4096 + li * 64 + ((g4 ^ ((0u - (li >> 2)) & 3)) << 4);
    u32 aw[4][2];
#pragma unroll
    for (int cc = 0; cc < 4; ++cc)
#pragma unroll
        for (int t2 = 0; t2 < 2; ++t2) {
            const u32 q = li >> 2, pp = lane & 3;
            aw[cc][t2] = 8192 + 256 * (8 * g4 + 4 * t2 + q)
                       + 16 * ((2 * (4 * wn + cc) + (pp >> 1)) ^ ((q << 2) | ((2 * g4 + t2) & 3))) + 8 * (pp & 1);
        }
    f32x4 acc[4][4], keep[4][4];
    auto readf = [&](u32 t, Frags& fr) {
        const LDSP char* sl = (const LDSP char*)smem + (t & 7) * 16384;
#pragma unroll
        for (int f = 0; f < 4; ++f) fr.x[f] = *(const LDSP bf16x8*)(sl + ax + f * 1024);
#pragma unroll
        for (int cc = 0; cc < 4; ++cc) {
            s16x4 lo = __builtin_amdgcn_ds_read_tr16_b64_v4i16((LDSP s16x4*)(sl + aw[cc][0]));
            s16x4 hi = __builtin_amdgcn_ds_read_tr16_b64_v4i16((LDSP s16x4*)(sl + aw[cc][1]));
            fr.w[cc] = __builtin_bit_cast(bf16x8, __builtin_shufflevector(lo, hi, 0, 1, 2, 3, 4, 5, 6, 7));
        }
    };
    auto mma = [&](const Frags& fr) {
#pragma unroll
        for (int c = 0; c < 4; ++c)
#pragma unroll
            for (int f = 0; f < 4; ++f)
                acc[c][f] = __builtin_amdgcn_mfma_f32_16x16x32_bf16(fr.w[c], fr.x[f], acc[c][f], 0, 0, 0);
    };
    auto interleave = [&]() {
#pragma unroll
        for (int i = 0; i < 4; ++i) {
            __builtin_amdgcn_sched_group_barrier(0x8, 1, 0);
            __builtin_amdgcn_sched_group_barrier(0x2, 2, 0);
            __builtin_amdgcn_sched_group_barrier(0x100, 2, 0);
        }
#pragma unroll
        for (int i = 0; i < 4; ++i) {
            __builtin_amdgcn_sched_group_barrier(0x8, 1, 0);
            __builtin_amdgcn_sched_group_barrier(0x2, 1, 0);
            __builtin_amdgcn_sched_group_barrier(0x100, 1, 0);
        }
        __builtin_amdgcn_sched_group_barrier(0x8, 3, 0);
        __builtin_amdgcn_sched_group_barrier(0x100, 1, 0);
        __builtin_amdgcn_sched_group_barrier(0x8, 5, 0);
        __builtin_amdgcn_sched_barrier(0);
    };
#pragma unroll
    for (int c = 0; c < 4; ++c)
#pragma unroll
        for (int f = 0; f < 4; ++f) acc[c][f] = f32x4{0.f, 0.f, 0.f, 0.f};
    u32 avail = 0;
    u32x4 peek = {0, 0, 0, 0};
    auto need = [&](u32 t) {
        avail = uni(min(min(peek[0], peek[1]), min(peek[2], peek[3])));
        while (__builtin_expect(avail <= t, 0)) {
            const u32x4 f = *(volatile LDSP u32x4*)ctl;
            avail = uni(min(min(f[0], f[1]), min(f[2], f[3])));
            if (avail <= t) __builtin_amdgcn_s_sleep(1);
        }
        asm volatile("" ::: "memory");
    };
    auto look = [&]() {
        peek = *(volatile LDSP u32x4*)ctl;
    };
    auto retire = [&](u32 t) {
        asm volatile("s_waitcnt lgkmcnt(0)" ::: "memory");
        ctl[4 + wave] = t + 1;
    };
    Frags fa, fb;
    need(0);
    readf(0, fa);
    auto steps = [&](u32 t0, u32 t1) {
        for (u32 t = t0; t < t1; t += 2) {
            retire(t);
            need(t + 1);
            readf(t + 1, fb);
            look();
            mma(fa);
            interleave();
            retire(t + 1);
            need(t + 2);
            readf(t + 2, fa);
            look();
            mma(fb);
            interleave();
        }
    };
    auto rs = __builtin_amdgcn_make_buffer_rsrc((void*)myws, 0, WS_NEED, 0x00020000);
    auto pack_acc = [&](u32x4 (&pk)[8], bool nan) {
#pragma unroll
        for (int f = 0; f < 4; ++f)
#pragma unroll
            for (int P = 0; P < 2; ++P) {
                const f32x4 lo = acc[2 * P][f], hi = acc[2 * P + 1][f];
                pk[f * 2 + P] = u32x4{pack2nz(lo[0], lo[1]), pack2nz(lo[2], lo[3]), pack2nz(hi[0], hi[1]), pack2nz(hi[2], hi[3])};
                if (nan) pk[f * 2 + P] = u32x4{0x7fc07fc0u, 0x7fc07fc0u, 0x7fc07fc0u, 0x7fc07fc0u};
            }
    };
    auto clear = [&](u32 tile) {
#pragma unroll
        for (int c = 0; c < 8; ++c)
            __builtin_amdgcn_raw_buffer_store_b128(u32x4{0, 0, 0, 0}, rs, lane * 16 + c * 1024,
                                                   OFF_DATA + (~call & 1) * PAR_BYTES + REL_CP + (tile * 4 + wave) * PIECE, AUX_SYS);
    };
    steps(0, T0);
    if (twice) {
        clear(m_d >= 4 && a_cb < PRZ ? (m_d - 4) * 8 + a_cb : m_d * 32 + ((ga ^ m_d) & 1) * 16 + a_cb);
        u32x4 pk[8];
        pack_acc(pk, uni(ctl[12]) != 0);
        const u32 e = (m_d * 8 + a_cb) * 4 + wave;
#pragma unroll
        for (int u = 0; u < 8; ++u)
            __builtin_amdgcn_raw_buffer_store_b128(pk[u], rs, lane * 16 + u * 1024, cbase - REL_CP + REL_IN + e * PIECE, AUX_SYS);
        wait_vm<0>();
        if (lane == 0) st_sys(myws + OFF_LF + e * 64, call);
    }
#pragma unroll
    for (int c = 0; c < 4; ++c)
#pragma unroll
        for (int f = 0; f < 4; ++f) { keep[c][f] = acc[c][f]; acc[c][f] = f32x4{0.f, 0.f, 0.f, 0.f}; }
    auto absorb = [&](u32 slot) {
        u32x4 v[8];
        u32 m = ~0u;
#pragma unroll
        for (int c = 0; c < 8; ++c) v[c] = __builtin_amdgcn_raw_buffer_load_b128(rs, lane * 16 + c * 1024, slot, AUX_SYS);
#pragma unroll
        for (int c = 0; c < 8; ++c) m = min(m, min(min(v[c][0], v[c][1]), min(v[c][2], v[c][3])));
        if (!__all(m != 0)) return false;
#pragma unroll
        for (int c = 0; c < 8; ++c) {
            const u32x4 a = v[c];
            acc[2 * (c & 1)][c >> 1] += f32x4{bf_lo(a[0]), bf_hi(a[0]), bf_lo(a[1]), bf_hi(a[1])};
            acc[2 * (c & 1) + 1][c >> 1] += f32x4{bf_lo(a[2]), bf_hi(a[2]), bf_lo(a[3]), bf_hi(a[3])};
        }
        return true;
    };
    const bool sums = m_d >= 4 && m_cb < PRZ && !nomain;
    const u32 in_slot = cbase + ((is_p ? (1 | m_d) * 32 + m_h * 16 + m_cb : (m_d - 4) * 8 + m_cb) * 4 + wave) * PIECE;
    bool taken = false;
#pragma clang loop unroll(disable)
    for (u32 t = T0, te = T - EARLY; t < T; t = te, te = T) {
        steps(t, te);
        if (te < T && sends) clear(m_d * 32 + ((m_h ^ m_d) & 1) * 16 + m_cb);
        if ((is_p || sums) && !taken) taken = absorb(in_slot);
    }
    bool bad = uni(ctl[12]) != 0;

    const u64 t0 = __builtin_amdgcn_s_memrealtime();
    u32 spin = 0;
    auto give_up = [&]() {
        asm volatile("" ::: "memory");
        if (__builtin_expect((++spin & 63u) == 0, 0) &&
            (uni(ld_agent(tmo)) == call || __builtin_amdgcn_s_memrealtime() - t0 > DEADLINE_TICKS)) {
            st_agent(tmo, call);
            bad = true;
        }
        return bad;
    };
    if (sends) {
        if (sums && !taken) {
            while (!bad) {
                const u32 probe = __builtin_amdgcn_raw_buffer_load_b32(rs, 7 * 1024, in_slot, AUX_SYS);
                if (__all(probe != 0) || give_up()) break;
                __builtin_amdgcn_s_sleep(POLL_SLEEP);
            }
            while (!absorb(in_slot) && !give_up()) { }
        }
        auto rd = __builtin_amdgcn_make_buffer_rsrc((void*)cws, 0, WS_NEED, 0x00020000);
        const u32 ci = (m_d * 32 + m_h * 16 + m_cb) * 4 + wave;
        u32x4 pk[8];
        pack_acc(pk, bad);
        while (uni(ctl[13]) == 0) __builtin_amdgcn_s_sleep(1);
        wait_vm<0>();
#pragma unroll
        for (int u = 0; u < 8; ++u) {
            wait_vm<C_WIN - 1>();
            __builtin_amdgcn_raw_buffer_store_b128(pk[u], rd, lane * 16 + u * 1024, cbase + ci * PIECE, AUX_SYS);
            if (u == C_REL && wave == 0 && lane == 0) add_agent(tok, C_SHARE);
        }
    } else if (is_p) {
#pragma unroll
        for (int c = 0; c < 4; ++c)
#pragma unroll
            for (int f = 0; f < 4; ++f) keep[c][f] = acc[c][f];
    }

    if (is_p || has_small) {
        const u32 db = ((o_ro >> 1) ^ gb) << 1;
        const bool one = is_p || o_cb < PRZ;
        const u32 l0 = is_p ? (1 | db) : (4 | db | (one ? ga ^ (o_ro & 1) ^ 1 : 0));
        const u32 tile = (o_ro & 1) * 16 + o_cb;
        auto ro = __builtin_amdgcn_make_buffer_rsrc((void*)pout, 0, IN_BYTES, 0x00020000);
        const u32 slot0 = cbase + ((l0 * 32 + tile) * 4 + wave) * PIECE;
        const u32 ocol = (is_p ? gc : 1 - gc) * 2048 + o_cb * 128 + wn * 64;
        const u32 obase = (o_ro * 128 + wm * 64) * WROW + ocol * 2;
        const u32 vo = (lane >> 2) * WROW + (lane & 3) * 16;
        const u32 osrc = ((((lane >> 1) & 1) | ((lane & 1) << 1)) * 16 + (lane >> 2)) * 4;
        auto put = [&](const f32x4& lo, const f32x4& hi, u32 c) {
            auto r0 = __builtin_amdgcn_permlane16_swap(pack2(lo[0], lo[1]), pack2(hi[0], hi[1]), false, false);
            auto r1 = __builtin_amdgcn_permlane16_swap(pack2(lo[2], lo[3]), pack2(hi[2], hi[3]), false, false);
            u32x4 o;
            o[0] = __builtin_amdgcn_ds_bpermute(osrc, r0[0]); o[1] = __builtin_amdgcn_ds_bpermute(osrc, r1[0]);
            o[2] = __builtin_amdgcn_ds_bpermute(osrc, r0[1]); o[3] = __builtin_amdgcn_ds_bpermute(osrc, r1[1]);
            if (bad) o = u32x4{0x7fc07fc0u, 0x7fc07fc0u, 0x7fc07fc0u, 0x7fc07fc0u};
            __builtin_amdgcn_raw_buffer_store_b128(o, ro, vo, obase + (c >> 1) * 16 * WROW + (c & 1) * 64, AUX_OUT);
        };
        if (is_p && taken) {
#pragma unroll
            for (int c = 0; c < 8; ++c) put(keep[2 * (c & 1)][c >> 1], keep[2 * (c & 1) + 1][c >> 1], c);
        } else {
        auto rb = __builtin_amdgcn_make_buffer_rsrc((void*)myws, 0, one ? 0 : WS_NEED, 0x00020000);
        const u32 skip_b = one ? ~0u : 0u;
        u32x4 pa[2], pb[2], na[2], nb[2];
        auto get = [&](u32x4 (&a)[2], u32x4 (&b)[2], u32 c) {
#pragma unroll
            for (int i = 0; i < 2; ++i) {
                a[i] = __builtin_amdgcn_raw_buffer_load_b128(rs, lane * 16 + i * 1024, slot0 + (c & 7) * 1024, AUX_SYS);
                b[i] = __builtin_amdgcn_raw_buffer_load_b128(rb, lane * 16 + i * 1024, slot0 + 128 * PIECE + (c & 7) * 1024, AUX_SYS);
            }
        };
        get(pa, pb, 0);
        __builtin_amdgcn_s_waitcnt(0x0F70);
        auto low = [](const u32x4& v) { return min(min(v[0], v[1]), min(v[2], v[3])); };
        auto there = [&]() { return __all(min(min(low(pa[0]), low(pa[1])), min(low(pb[0]), low(pb[1])) | skip_b) != 0); };
        if (!there()) {
            while (!bad) {
                const u32 probe = __builtin_amdgcn_raw_buffer_load_b32(rs, one ? 0 : (lane & 1) * 128 * PIECE, slot0, AUX_SYS);
                if (__all(probe != 0) || give_up()) break;
                __builtin_amdgcn_s_sleep(POLL_SLEEP);
            }
        }
#pragma unroll
        for (int c = 0; c < 8; c += 2) {
            f32x4 sum[4];
#pragma unroll
            for (int i = 0; i < 4; ++i) sum[i] = keep[i][c >> 1];
            while (!there() && !give_up()) {
                get(pa, pb, c);
                __builtin_amdgcn_s_waitcnt(0x0F70);
            }
            get(na, nb, c + 2);
#pragma unroll
            for (int i = 0; i < 2; ++i) {
                const u32x4 a = pa[i], b = pb[i];
                sum[2 * i] += f32x4{bf_lo(a[0]), bf_hi(a[0]), bf_lo(a[1]), bf_hi(a[1])};
                sum[2 * i + 1] += f32x4{bf_lo(a[2]), bf_hi(a[2]), bf_lo(a[3]), bf_hi(a[3])};
                sum[2 * i] += f32x4{bf_lo(b[0]), bf_hi(b[0]), bf_lo(b[1]), bf_hi(b[1])};
                sum[2 * i + 1] += f32x4{bf_lo(b[2]), bf_hi(b[2]), bf_lo(b[3]), bf_hi(b[3])};
            }
            put(sum[0], sum[1], c);
            put(sum[2], sum[3], c + 1);
            if (c == 6) break;
            __builtin_amdgcn_s_waitcnt(0x0F72);
            pa[0] = na[0]; pa[1] = na[1]; pb[0] = nb[0]; pb[1] = nb[1];
        }
        }
    }
    if (threadIdx.x == 0) st_agent(ctr, call);
}

static void build_schedule(Params& p) {
    static const int CL[6] = {1, 3, 4, 5, 6, 7};
    auto steps_of = [](int rel) {
        const bool shared = rel != 2 && rel != 3;
        const u32 skip = ((rel & 5) ? 1u << rel : 0u) | (shared ? 2u : 0u);
        return skip << 21 | (u32)(shared || (rel & 5) == 0) << 29;
    };
    for (int n = 0; n < 64; ++n) {
        const int i = n & 31, x = n >> 5;
        const int rel = ((i >> 3) & 1) * 2, cb = (i >> 4) * 8 + x * 4 + ((i >> 1) & 3);
        p.role[i * 8 + x * 2] = (u32)(rel | (n & 1) << 3 | cb << 4) | steps_of(rel);
    }
    const int ga = (int)p.rank & 1;
    for (int li = 0; li < 6; ++li) {
        const int l = CL[li];
        for (int i = 0; i < 32; ++i) {
            u32 e = (u32)l | steps_of(l);
            const int n = i & 7;
            if (i < 8) e |= 1u << 31 | 1u << 14 | (u32)(li * 8 + n) << 15;
            else if (i < 16) {
                const int cb = l >= 4 ? (n + (int)PRZ) & 7 : n;
                e |= (u32)((1 - ga) << 3 | cb << 4 | n << 8);
            } else {
                e |= (u32)((n & 1) << 3 | (i >> 1) << 4 | (i - 8) << 8);
                if (i >= 24) e |= 1u << 30 | (u32)n << 15;
                else if (n < 3 && 48 + li * 3 + n < 64) e |= 1u << 14 | (u32)(48 + li * 3 + n) << 15;
            }
            p.role[i * 8 + l] = e;
        }
    }
    auto detour = [](int l, int e) {
        const int blk = e & 31, j = e >> 5, pr = (31 - blk) >> 1;
        if (l == 1) return pr == 1 && (7 - j) * 2 < FWD1;
        if (l < 3 || blk < 16) return false;
        if (l == 3) return pr * 16 + j * 2 < FWD3;
        if ((j >> 2) != (l >= 6)) return false;
        return pr * 8 + (j & 3) * 2 < FWD;
    };
    static const int lag[2][8] = {{0, 0, 12, 24, 24, 28, 6, 10}, {0, 0, 36, 52, 64, 120, 116, 12}};
    int when[8][256];
    for (int l = 0; l < 8; ++l) {
        for (int e = 0; e < 256; ++e) when[l][e] = 0;
        for (int w = 0; w < 8; ++w) p.plist[l][w][0] = p.plist[l][w][1] = p.plist[l][w][2] = p.plist[l][w][3] = 0;
    }
    for (int l = 1; l < 8; ++l) {
        int lst[384], key[384], n = 0;
        auto put = [&](int e, int k) {
            int i = n++;
            for (; i > 0 && key[i - 1] > k; --i) { key[i] = key[i - 1]; lst[i] = lst[i - 1]; }
            key[i] = k; lst[i] = e;
        };
        for (int e = 0; e < 256; ++e) {
            const int blk = e & 31, j = e >> 5, rel = blk >> 1;
            if (blk < 16 && ((steps_of(rel) >> 21 >> l) & 1)) continue;
            if (detour(l, e)) continue;
            int lg = blk >= 16 ? lag[l == 2][col_class(blk - 16)] : lag[l == 2][(rel & 5) ? 7 : rel ? 5 : 6];
            if (detour(l ^ 2, e)) lg = j >= 4 ? -2 : 6;
            const bool express = blk >= 16 ? blk < 24 : ((rel & 5) && (blk & 1) == ((ga ^ l) & 1));
            const bool first = blk >= 16 ? blk - 16 < (int)PRZ : rel >= 4;
            put(e, (express ? (first ? j : j * 2 + 2) : j * 4 + lg) * 256 + blk);
        }
        if (l == 2)
            for (int s = 1; s < 8; ++s)
                for (int e = 0; e < 256; ++e)
                    if (detour(s ^ 2, e)) put(s << 8 | e, (84 + (e >> 5) * 2 - ((31 - (e & 31)) >> 1) * 6) * 256 + s);
        int cut = n;
        for (int i = n - 1; i >= 0; --i) if (lst[i] >= 256) cut = i & ~7;
        for (int i = 0; i < n; ++i) {
            when[lst[i] >= 256 ? (lst[i] >> 8) ^ 2 : l][lst[i] & 255] = i;
            if (i < cut) p.plist[l][i & 7][i >> 6] |= (u64)lst[i] << (8 * ((i >> 3) & 7));
            else if (i - cut < 128) p.plist[0][(i - cut) & 7][(i - cut) >> 5] |= (u64)lst[i] << (16 * (((i - cut) >> 3) & 3));
        }
        p.npieces[l] = (u32)cut;
        if (l == 2) p.npieces[0] = (u32)(n - cut);
    }
    for (int c = 0; c < 12; ++c) {
        static const int rep[5] = {19, 23, 27, 29, 31};
        int key[64], n = 0;
        for (int i = 0; i < 64; ++i) p.order[c][i] = 0;
        for (int b = 8; b < 64; ++b) {
            const int l = b >> 3, j = b & 7;
            const int x0 = when[l][j << 5 | (l == 7 ? 12 : 14)], x1 = when[l][j << 5 | (l == 7 ? 13 : 15)];
            const int tw = when[l][j << 5 | (c == 10 ? 23 : rep[c % 5])];
            const int tx = c == 11 ? when[l][j << 5 | 1] : c == 10 ? (x0 < x1 ? x0 : x1) : c >= 5 ? when[l][j << 5 | 5] : (x0 > x1 ? x0 : x1);
            const int k = (tw > tx ? tw : tx) * 8 + l;
            int i = n++;
            for (; i > 0 && key[i - 1] > k; --i) { key[i] = key[i - 1]; p.order[c][i] = p.order[c][i - 1]; }
            key[i] = k; p.order[c][i] = (unsigned char)b;
        }
    }
}

extern "C" void kernel_launch(int rank, int nranks, ncclComm_t comm,
                              hipStream_t stream,
                              void* const* d_in, const int* in_sizes, int n_in,
                              void* d_out, int out_size,
                              const int* in_dtypes, int out_dtype,
                              void* const* d_out_all,
                              void* const* d_ws_all, size_t ws_bytes) {
    if (nranks != NR || (unsigned)rank >= (unsigned)NR || n_in < 2 || ws_bytes < WS_NEED ||
        in_sizes[0] != 4096 * 512 || in_sizes[1] != 512 * 4096 || out_size != 512 * 4096) {
        fprintf(stderr, "kernel_launch: unexpected problem shape\n");
        return;
    }
    Params p;
    p.x = (const char*)d_in[0];
    p.w = (const char*)d_in[1];
    p.out = (char*)d_out;
    u32 id = 0;
    while (rank_of(id) != (u32)rank) ++id;
    for (u32 d = 0; d < NR; ++d) p.peer[d] = (char*)d_ws_all[rank_of(id ^ d)];
    p.self = p.peer[0];
    p.rank = id;
    p.pad = 0;
    build_schedule(p);
    if (p.npieces[0] > 128) { fprintf(stderr, "kernel_launch: schedule overflow\n"); return; }
    hipLaunchKernelGGL(gemm_rs_kernel, dim3(256), dim3(512), 0, stream, p);
    HIP_CHECK(hipGetLastError());
}
```
